```python
import numpy as np

kernel_name = "gpu_gemm_rs_m8192_k8192_n8192_bf16_silu"


ABSMAX_RTOL = 0.008

COMPUTE_DTYPE = "bfloat16"
EPILOGUE = "silu"
IN_QUANT = None
OUT_QUANT = None
IN_DTYPES = {"x": "bfloat16", "w_mat": "bfloat16"}
WORLD_SIZE = 8

def _to_bf16(x):
    u = np.ascontiguousarray(x, dtype=np.float32).view(np.uint32)
    return ((u + 0x7FFF + ((u >> 16) & 1)) & 0xFFFF0000).view(np.float32)

def setup_inputs():

    x = _to_bf16(np.random.randn(8192, 8192).astype(np.float32))
    w_mat = _to_bf16((np.random.randn(8192, 8192) / np.sqrt(8192)).astype(np.float32))
    return {"x": x, "w_mat": w_mat}

def _epilogue(y):
    return (y / (1.0 + np.exp(-np.clip(y, -60.0, 60.0)))).astype(np.float32)

def reference(x, w_mat):
    y = (x @ w_mat).astype(np.float32)
    return _epilogue(y)
```

```cpp
#include <hip/hip_runtime.h>
#include <rccl/rccl.h>
#include <cstdio>

#define HIP_CHECK(expr) do { hipError_t _e = (expr); if (_e != hipSuccess) { \
    fprintf(stderr, "HIP error %s at %s:%d\n", hipGetErrorString(_e), __FILE__, __LINE__); \
    return; } } while (0)

namespace {

#define LAS __attribute__((address_space(3)))
#define GAS __attribute__((address_space(1)))
typedef short bf16x8 __attribute__((ext_vector_type(8)));
typedef short bf16x4 __attribute__((ext_vector_type(4)));
typedef float f32x4 __attribute__((ext_vector_type(4)));
typedef unsigned u32x4 __attribute__((ext_vector_type(4)));
typedef GAS unsigned gu32;

constexpr int NRANK = 8;
constexpr unsigned MiB = 1u << 20;
constexpr unsigned XROW = 2048;
constexpr unsigned WROW = 16384;
constexpr unsigned IN_BYTES = 16 * MiB;
constexpr unsigned OUT_BYTES = 16 * MiB;
constexpr unsigned TILE_BYTES = 128 * 1024;
constexpr unsigned WS_CALL = 0;
constexpr unsigned WS_TMO = 4096;
constexpr unsigned WS_PROG = 8192;
constexpr unsigned WS_READY = 16384;
constexpr unsigned WS_XR = 1 * MiB;
constexpr unsigned XARR = 8 * MiB, XR_PAR = 24 * MiB;
constexpr unsigned WS_WR = WS_XR + 48 * MiB;
constexpr unsigned WS_LP = WS_WR + 48 * MiB;
constexpr unsigned WS_RP = WS_LP + 24 * MiB;
constexpr unsigned RP_PAR = 24 * MiB;
constexpr unsigned WS_END = WS_RP + 48 * MiB;
constexpr unsigned WS_DBG = WS_END;
constexpr unsigned WS_NEED = WS_END + 1 * MiB;

constexpr int HTB = 16384;
constexpr int LDS_BYTES = 8 * HTB;
constexpr unsigned long long DEADLINE_TICKS = 50000000ull;
constexpr int MAXITEMS = 96;

struct Params {
    const char* x; const char* w; char* out;
    char* ws[NRANK];
    unsigned ws_bytes; int rank;
    unsigned char nitems[8], ninput[8];
    unsigned char item[8][MAXITEMS];
    unsigned char pos[8][32];
    unsigned char tpos[192];
    unsigned short job[256][2];
};

__device__ __forceinline__ unsigned cvt_pk_bf16(float lo, float hi) {
    unsigned r; asm("v_cvt_pk_bf16_f32 %0, %1, %2" : "=v"(r) : "v"(lo), "v"(hi)); return r;
}
__device__ __forceinline__ float bf_lo(unsigned u) { return __uint_as_float(u << 16); }
__device__ __forceinline__ float bf_hi(unsigned u) { return __uint_as_float(u & 0xffff0000u); }
__device__ __forceinline__ float silu(float y) {
    const float c = fminf(fmaxf(y, -60.0f), 60.0f);
    return y * __builtin_amdgcn_rcpf(1.0f + __expf(-c));
}
__device__ __forceinline__ unsigned ld_sys(const char* p) {
    return __hip_atomic_load((const gu32*)p, __ATOMIC_RELAXED, __HIP_MEMORY_SCOPE_SYSTEM);
}
__device__ __forceinline__ unsigned ld_agent(const char* p) {
    return __hip_atomic_load((const gu32*)p, __ATOMIC_RELAXED, __HIP_MEMORY_SCOPE_AGENT);
}
__device__ __forceinline__ unsigned grow_of(int g, int lr) {
    if (lr < 4) return (unsigned)((g ^ 1) * 1024 + lr * 256);
    if (lr >= 12) return (unsigned)(g * 1024 + (lr - 12) * 256);
    const int r = lr - 4;
    return (unsigned)((g ^ (4 + (r & 3))) * 1024 + ((g >> 1) & 1) * 512 + (r >> 2) * 256);
}
__device__ __forceinline__ bool give_up(char* ws, unsigned call, unsigned long long t0) {
    const unsigned t = __builtin_amdgcn_readfirstlane(ld_agent(ws + WS_TMO));
    if (t == call) return true;
    if (__builtin_amdgcn_s_memrealtime() - t0 > DEADLINE_TICKS) {
        __hip_atomic_store((gu32*)(ws + WS_TMO), call, __ATOMIC_RELAXED, __HIP_MEMORY_SCOPE_AGENT);
        return true;
    }
    return false;
}

#define STAMP(i) do {} while (0)

#define WAIT_V(n) asm volatile("s_waitcnt vmcnt(" #n ")" ::: "memory")
#define WAIT_L(n) asm volatile("s_waitcnt lgkmcnt(" #n ")" ::: "memory")
#define BAR asm volatile("s_barrier" ::: "memory")
#define SCHED __builtin_amdgcn_sched_barrier(0)

__device__ __forceinline__ void comm_role(const Params& p, const int cid, const unsigned call) {
    const int g = p.rank, d = (cid >> 2) + 1, peer = g ^ d;
    if ((unsigned)peer >= (unsigned)NRANK) return;
    char* ws = p.ws[g];
    const int lane = threadIdx.x & 63;
    const int wv = (cid & 3) * 8 + __builtin_amdgcn_readfirstlane(threadIdx.x >> 6);
    const unsigned par = call & 1u;
    const int nit = p.nitems[d], nin = p.ninput[d];
    const int ps3 = peer & 1, ps1 = (peer >> 2) & 1, slot = d - 2;
    const auto rx = __builtin_amdgcn_make_buffer_rsrc((void*)p.x, 0, IN_BYTES, 0x00020000);
    const auto rw = __builtin_amdgcn_make_buffer_rsrc((void*)p.w, 0, IN_BYTES, 0x00020000);
    const auto rl = __builtin_amdgcn_make_buffer_rsrc((void*)ws, 0, WS_END, 0x00020000);
    const auto rd = __builtin_amdgcn_make_buffer_rsrc((void*)p.ws[peer], 0, WS_END, 0x00020000);
    gu32* flag = (gu32*)(p.ws[peer] + WS_PROG + d * 256 + wv * 4);
    const unsigned lane_lin = (unsigned)lane * 16u, lane_w = (unsigned)(lane >> 5) * WROW + (unsigned)(lane & 31) * 16u;
    const unsigned long long t0 = __builtin_amdgcn_s_memrealtime();
    int nready = 0;
    u32x4 v[4];
    unsigned dofs = 0, dstr = 0;
    for (int i = 0; i <= nit; ++i) {
        u32x4 nv[4]; unsigned ndofs = 0, nstr = 0;
        bool stored = false;
        if (i < nit) {
            const unsigned it = p.item[d][i];
            if (i >= nin && i - nin >= nready) {
                if (i > 0) {
#pragma unroll
                    for (int u = 0; u < 4; ++u) __builtin_amdgcn_raw_buffer_store_b128(v[u], rd, dofs + u * dstr, 0, 17);
                    WAIT_V(0);
                    if (lane == 0) __hip_atomic_store(flag, call * 128u + (unsigned)i, __ATOMIC_RELAXED, __HIP_MEMORY_SCOPE_SYSTEM);
                    stored = true;
                }
                for (;;) {
                    const unsigned f = ld_agent(ws + WS_READY + d * 256 + lane * 4);
                    const unsigned long long notyet = __ballot(f != call);
                    nready = notyet ? (int)__builtin_ctzll(notyet) : 64;
                    if (i - nin < nready) break;
                    if (give_up(ws, call, t0)) return;
                    __builtin_amdgcn_s_sleep(8);
                }
            }
            if (i < nin) {
                const unsigned blk = (it >> 1) & 15u, hh = it & 1u;
                if (it & 64u) {
                    nstr = 2u * WROW;
                    const unsigned so = ((unsigned)ps3 * 512u + hh * 256u) * WROW + (unsigned)ps1 * 8192u + blk * 512u + (unsigned)wv * 4u * nstr + lane_w;
                    ndofs = WS_WR + ((unsigned)slot * 512u + hh * 256u) * WROW + par * 8192u + blk * 512u + (unsigned)wv * 4u * nstr + lane_w;
#pragma unroll
                    for (int u = 0; u < 4; ++u) nv[u] = __builtin_amdgcn_raw_buffer_load_b128(rw, so + u * nstr, 0, 0);
                } else {
                    nstr = XROW;
                    const unsigned so = (grow_of(peer, (int)blk) + hh * 128u + (unsigned)wv * 4u) * XROW + (unsigned)ps3 * 1024u + lane_lin;
                    ndofs = WS_XR + par * XR_PAR + (unsigned)(slot >> 1) * XARR + (blk * 256u + hh * 128u + (unsigned)wv * 4u) * XROW + (unsigned)(slot & 1) * 1024u + lane_lin;
#pragma unroll
                    for (int u = 0; u < 4; ++u) nv[u] = __builtin_amdgcn_raw_buffer_load_b128(rx, so + u * nstr, 0, 0);
                }
            } else {
                nstr = 1024u;
                const unsigned lr = it >> 4, lc = it & 15u;
                const unsigned dt = lr < 4u ? it : 64u + (unsigned)(g & 1) * 64u + ((unsigned)((g >> 1) & 1) * 2u + ((lr - 4u) >> 2)) * 16u + lc;
                const unsigned so = WS_LP + it * TILE_BYTES + (unsigned)wv * 4096u + lane_lin;
                ndofs = WS_RP + par * RP_PAR + dt * TILE_BYTES + (unsigned)wv * 4096u + lane_lin;
#pragma unroll
                for (int u = 0; u < 4; ++u) nv[u] = __builtin_amdgcn_raw_buffer_load_b128(rl, so + u * nstr, 0, 16);
            }
        }
        if (i > 0 && !stored) {
#pragma unroll
            for (int u = 0; u < 4; ++u) __builtin_amdgcn_raw_buffer_store_b128(v[u], rd, dofs + u * dstr, 0, 17);
            if (i < nit) {
                WAIT_V(4);
                if (i > 1 && lane == 0) __hip_atomic_store(flag, call * 128u + (unsigned)(i - 1), __ATOMIC_RELAXED, __HIP_MEMORY_SCOPE_SYSTEM);
            }
        }
#pragma unroll
        for (int u = 0; u < 4; ++u) v[u] = nv[u];
        dofs = ndofs; dstr = nstr;
    }
    WAIT_V(0);
    if (lane == 0) __hip_atomic_store(flag, call * 128u + (unsigned)nit, __ATOMIC_RELAXED, __HIP_MEMORY_SCOPE_SYSTEM);
}

__global__ __launch_bounds__(512, 2) void gemm_rs_kernel(const Params p) {
    extern __shared__ __attribute__((aligned(16))) unsigned char shm[];
    LAS unsigned char* lds = (LAS unsigned char*)shm;
    const int g = p.rank;
    char* ws = p.ws[g];
    const int bid = blockIdx.x;
    const unsigned call = __builtin_amdgcn_readfirstlane(ld_agent(ws + WS_CALL + bid * 4)) + 1u;
    STAMP(0);
    if (bid < 32) {
        if (bid < 28) comm_role(p, bid, call);
    } else {
        const int tid = threadIdx.x, wid = __builtin_amdgcn_readfirstlane(tid >> 6), lane = tid & 63;
        const int wr = wid >> 2, wc = wid & 3, fr = lane & 15, fq = lane >> 4;
        const unsigned par = call & 1u;
        const int s1 = (g >> 2) & 1;
        unsigned voffA, voffB;
        { const int r = wid * 8 + (lane >> 3); voffA = (unsigned)r * XROW + (unsigned)(((lane & 7) ^ ((r >> 1) & 7)) * 16); }
        { const int r = wid * 4 + (lane >> 4); voffB = (unsigned)r * WROW + (unsigned)(((lane & 15) ^ (((r & 3) << 2) | (((r >> 3) & 1) << 1))) * 16); }
        const unsigned ldsw = (unsigned)(size_t)lds + (unsigned)wid * 1024u;
        const int aoff0 = (wr * 64 + fr) * 128 + ((fq ^ (fr >> 1)) * 16), aoff1 = aoff0 ^ 64;
        int boff0, boff1;
        { const int q = (lane & 15) >> 2, pp = lane & 3, r = 8 * fq + q;
          boff0 = 4 * HTB + r * 256 + (((wc * 4 + (pp >> 1)) ^ ((q << 2) | ((fq & 1) << 1))) * 16) + (pp & 1) * 8; boff1 = boff0 ^ 32; }
        asm volatile("" : "+v"(boff0), "+v"(boff1));
#define SA(b, h) (((b) * 2 + (h)) * HTB)
#define SB(b, h) (((b) * 2 + (h)) * HTB)
#define DMA2(rs, voff, so, l0, o0, o1) do { unsigned _t; asm volatile( \
        "s_add_u32 m0, %3, %5\n\ts_add_u32 %0, %4, %7\n\tbuffer_load_dwordx4 %1, %2, %0 offen lds\n\t" \
        "s_add_u32 m0, %3, %6\n\ts_add_u32 %0, %4, %8\n\tbuffer_load_dwordx4 %1, %2, %0 offen lds" \
        : "=&s"(_t) : "v"(voff), "s"(rs), "s"(ldsw), "s"(so), "i"(l0), "i"((l0) + 8192), "i"(o0), "i"(o1) : "memory", "scc"); } while (0)
#define STAGE_A(b, h, so, odd) DMA2(ra, voffA, so, SA(b, h), (h) * 128 * XROW + (odd) * 128, ((h) * 128 + 64) * XROW + (odd) * 128)
#define STAGE_B(b, h, so, odd) DMA2(rb, voffB, so, 4 * HTB + SB(b, h), (h) * 256 + (odd) * 64 * WROW, (h) * 256 + ((odd) * 64 + 32) * WROW)
#define LDA(dst, b, h) do { _Pragma("unroll") for (int m = 0; m < 4; ++m) { \
        dst[m][0] = *(const LAS bf16x8*)(lds + SA(b, h) + aoff0 + m * 2048); dst[m][1] = *(const LAS bf16x8*)(lds + SA(b, h) + aoff1 + m * 2048); } } while (0)
#define TR(off) __builtin_amdgcn_ds_read_tr16_b64_v4i16((LAS bf16x4*)(lds + (off)))
#define LDB(dst, b, h) do { _Pragma("unroll") for (int k = 0; k < 2; ++k) { \
        dst[0][k] = __builtin_shufflevector(TR(SB(b, h) + boff0 + k * 8192), TR(SB(b, h) + boff0 + k * 8192 + 1024), 0, 1, 2, 3, 4, 5, 6, 7); \
        dst[1][k] = __builtin_shufflevector(TR(SB(b, h) + boff1 + k * 8192), TR(SB(b, h) + boff1 + k * 8192 + 1024), 0, 1, 2, 3, 4, 5, 6, 7); } } while (0)
#define PIN(ai, bj) asm volatile("" : "+v"(acc[ai][bj][0][0]), "+v"(acc[ai][bj][0][1]), "+v"(acc[ai][bj][1][0]), "+v"(acc[ai][bj][1][1]), \
        "+v"(acc[ai][bj][2][0]), "+v"(acc[ai][bj][2][1]), "+v"(acc[ai][bj][3][0]), "+v"(acc[ai][bj][3][1]))
#define MMA(ai, bj, At, Bt) do { __builtin_amdgcn_s_setprio(1); PIN(ai, bj); \
        _Pragma("unroll") for (int m = 0; m < 4; ++m) _Pragma("unroll") for (int n = 0; n < 2; ++n) _Pragma("unroll") for (int k = 0; k < 2; ++k) \
        acc[ai][bj][m][n] = __builtin_amdgcn_mfma_f32_16x16x32_bf16(Bt[n][k], At[m][k], acc[ai][bj][m][n], 0, 0, 0); \
        PIN(ai, bj); __builtin_amdgcn_s_setprio(0); } while (0)

        for (int jn = 0; jn < 2; ++jn) {
            const unsigned job = p.job[bid][jn];
            const int type = job >> 8, lr = (job >> 4) & 15, lc = job & 15;
            if (type == 0) break;
            f32x4 acc[2][2][4][2];
#pragma unroll
            for (int a = 0; a < 2; ++a)
#pragma unroll
                for (int b = 0; b < 2; ++b)
#pragma unroll
                    for (int m = 0; m < 4; ++m)
#pragma unroll
                        for (int n = 0; n < 2; ++n) acc[a][b][m][n] = (f32x4){0.f, 0.f, 0.f, 0.f};
            const int nph = type == 4 ? 1 : 2;
            bool late = false;
            for (int ph = 0; ph < nph; ++ph) {
                const char* abase; const char* bbase; unsigned a0, b0, nbytes; int nt, skip = 99;
                if (ph == 0) {
                    const unsigned grow = type == 4 ? (unsigned)(g * 1024 + lr * 256) : grow_of(g, lr);
                    const unsigned gcol = (unsigned)((type == 4 ? 1 - s1 : s1) * 4096 + lc * 256);
                    abase = p.x; bbase = p.w; nbytes = IN_BYTES;
                    a0 = grow * XROW; b0 = gcol * 2u; nt = 16;
                } else {
                    abase = ws; bbase = ws; nbytes = WS_END;
                    a0 = WS_XR + par * XR_PAR + (unsigned)lr * 256u * XROW; b0 = WS_WR + par * 8192u + (unsigned)lc * 512u;
                    nt = 48;
                    if (type == 2) { skip = ((lr - 4) & 3) + 2; nt = 40; }
                    STAMP(1 + jn * 6);
                    if (wid == 0) {
                        unsigned need[3];
#pragma unroll
                        for (int i = 0; i < 3; ++i) { const int sl = 2 * i + (lane >> 5);
                            const unsigned px = p.pos[sl + 2][lr], pw = p.pos[sl + 2][16 + lc];
                            need[i] = sl == skip ? 0u : call * 128u + (px > pw ? px : pw); }
                        const unsigned long long t0 = __builtin_amdgcn_s_memrealtime();
                        const char* fp = ws + WS_PROG + (2 + (lane >> 5)) * 256 + (lane & 31) * 4;
                        for (;;) {
                            const unsigned f0 = ld_sys(fp), f1 = ld_sys(fp + 512), f2 = ld_sys(fp + 1024);
                            if (__all(f0 >= need[0] && f1 >= need[1] && f2 >= need[2])) break;
                            if (give_up(ws, call, t0)) { late = true; break; }
                            __builtin_amdgcn_s_sleep(16);
                        }
                        __builtin_amdgcn_fence(__ATOMIC_ACQUIRE, "");
                        WAIT_V(0);
                    }
                    asm volatile("" ::: "memory"); BAR; asm volatile("" ::: "memory");
                    STAMP(2 + jn * 6);
                }
                bf16x8 At[4][2], B0[2][2], B1[2][2];
                const u32x4 ra = {(unsigned)(size_t)abase, (unsigned)((size_t)abase >> 32), nbytes, 0x00020000u};
                const u32x4 rb = {(unsigned)(size_t)bbase, (unsigned)((size_t)bbase >> 32), nbytes, 0x00020000u};
#define KOFF(t, oa, ob) do { const int _sg = (t) >> 3, _s = _sg + (_sg >= skip ? 1 : 0); \
                oa = a0 + (unsigned)(_s >> 1) * XARR + (unsigned)(_s & 1) * 1024u + (unsigned)((t) & 7) * 128u; \
                ob = b0 + (unsigned)(_s * 8 + ((t) & 7)) * (64u * WROW); } while (0)
                unsigned ca, cb;
                KOFF(0, ca, cb);
                STAGE_B(0, 0, cb, 0); STAGE_A(0, 0, ca, 0); STAGE_B(0, 1, cb, 0); STAGE_A(0, 1, ca, 0);
                if (wr == 1) BAR;
                WAIT_V(4); BAR;
                STAGE_B(1, 0, cb, 1); STAGE_A(1, 0, ca, 1); STAGE_B(1, 1, cb, 1);
                WAIT_V(6); BAR;
                for (int t = 0; t < nt; t += 2) {
                    const int tn = t + 2 < nt ? t + 2 : 0;
                    unsigned a2, b2;
                    KOFF(tn, a2, b2);
                    LDB(B0, 0, 0); SCHED; LDA(At, 0, 0); STAGE_A(1, 1, ca, 1);
                    WAIT_L(8); BAR; WAIT_L(0); MMA(0, 0, At, B0); BAR; SCHED;
                    LDB(B1, 0, 1); STAGE_B(0, 0, b2, 0);
                    BAR; WAIT_L(0); MMA(0, 1, At, B1); BAR;
                    LDA(At, 0, 1); STAGE_A(0, 0, a2, 0);
                    BAR; WAIT_L(0); MMA(1, 0, At, B0); BAR; SCHED;
                    STAGE_B(0, 1, b2, 0);
                    WAIT_V(6); BAR; MMA(1, 1, At, B1); BAR;
                    LDB(B0, 1, 0); SCHED; LDA(At, 1, 0); STAGE_A(0, 1, a2, 0);
                    WAIT_L(8); BAR; WAIT_L(0); MMA(0, 0, At, B0); BAR; SCHED;
                    LDB(B1, 1, 1); STAGE_B(1, 0, b2, 1);
                    BAR; WAIT_L(0); MMA(0, 1, At, B1); BAR;
                    LDA(At, 1, 1); STAGE_A(1, 0, a2, 1);
                    BAR; WAIT_L(0); MMA(1, 0, At, B0); BAR; SCHED;
                    STAGE_B(1, 1, b2, 1);
                    WAIT_V(6); BAR; MMA(1, 1, At, B1); BAR;
                    ca = a2;
                }
                WAIT_V(0);
                if (wr == 0) BAR;
                BAR;
            }
            STAMP(3 + jn * 6);
            int tz = threadIdx.x; asm volatile("" : "+v"(tz));
            const int elane = tz & 63;
            const unsigned toff = (unsigned)wid * 16384u + (unsigned)elane * 16u;
            const auto rl = __builtin_amdgcn_make_buffer_rsrc((void*)ws, 0, WS_END, 0x00020000);
            if (type <= 2) {
                const unsigned tl = (unsigned)(lr * 16 + lc);
                const unsigned base = WS_LP + tl * TILE_BYTES + toff;
#pragma unroll
                for (int a = 0; a < 2; ++a)
#pragma unroll
                    for (int b = 0; b < 2; ++b)
#pragma unroll
                        for (int m = 0; m < 4; ++m) {
                            const f32x4 v0 = acc[a][b][m][0], v1 = acc[a][b][m][1];
                            u32x4 o; o.x = cvt_pk_bf16(v0[0], v0[1]); o.y = cvt_pk_bf16(v0[2], v0[3]); o.z = cvt_pk_bf16(v1[0], v1[1]); o.w = cvt_pk_bf16(v1[2], v1[3]);
                            __builtin_amdgcn_raw_buffer_store_b128(o, rl, base + (unsigned)(((a * 2 + b) * 4 + m) * 1024), 0, 16);
                        }
                WAIT_V(0);
                BAR;
                if (tz == 0 && !late) {
                    const int d = lr < 4 ? 1 : 4 + ((lr - 4) & 3);
                    __hip_atomic_store((gu32*)(ws + WS_READY + d * 256 + p.tpos[tl] * 4), call, __ATOMIC_RELAXED, __HIP_MEMORY_SCOPE_AGENT);
                }
            } else {
                const int orow = type == 3 ? lr - 12 : lr;
                LAS unsigned* lflag = (LAS unsigned*)lds;
                if (wid == 0) {
                    int d; unsigned need;
                    if (type == 3) { d = 1; need = p.tpos[orow * 16 + lc] + 1u; }
                    else { const int h = ((1 - s1) << 2) | ((orow >> 1) << 1) | (elane >> 5); d = h ^ g;
                           need = p.ninput[d] + p.tpos[(4 + (orow & 1) * 4 + (d - 4)) * 16 + lc] + 1u; }
                    need += call * 128u;
                    const char* fp = ws + WS_PROG + d * 256 + (elane & 31) * 4;
                    const unsigned long long t0 = __builtin_amdgcn_s_memrealtime();
                    for (;;) {
                        const unsigned f = ld_sys(fp);
                        if (__all(f >= need)) break;
                        if (give_up(ws, call, t0)) break;
                        __builtin_amdgcn_s_sleep(8);
                    }
                    __builtin_amdgcn_fence(__ATOMIC_ACQUIRE, "");
                    const unsigned tm = ld_agent(ws + WS_TMO);
                    if (elane == 0) lflag[0] = tm == call ? 1u : 0u;
                    asm volatile("s_waitcnt vmcnt(0) lgkmcnt(0)" ::: "memory");
                }
                asm volatile("" ::: "memory"); BAR; asm volatile("" ::: "memory");
                STAMP(4 + jn * 6);
                const bool bad = __builtin_amdgcn_readfirstlane(lflag[0]) != 0u;
                const unsigned t1 = type == 3 ? (unsigned)(orow * 16 + lc) : 64u + (unsigned)(orow * 16 + lc);
                const unsigned base1 = bad ? WS_END : WS_RP + par * RP_PAR + t1 * TILE_BYTES + toff;
                const unsigned base2 = (bad || type == 3) ? WS_END : base1 + 64u * TILE_BYTES;
                const int efr = elane & 15, efq = elane >> 4;
                const auto ro = __builtin_amdgcn_make_buffer_rsrc((void*)p.out, 0, OUT_BYTES, 0x00020000);
                const unsigned ocol = (unsigned)((type == 3 ? s1 : 1 - s1) * 4096 + lc * 256 + wc * 32 + (efq & 1) * 16 + (efq >> 1) * 8);
                const unsigned obase = (unsigned)(orow * 256 + wr * 64 + efr) * WROW + ocol * 2u;
                u32x4 q1[2][4], q2[2][4];
#define LDQ(s, grp) do { _Pragma("unroll") for (int m = 0; m < 4; ++m) { \
                    q1[s][m] = __builtin_amdgcn_raw_buffer_load_b128(rl, base1 + (unsigned)(((grp) * 4 + m) * 1024), 0, 0); \
                    q2[s][m] = __builtin_amdgcn_raw_buffer_load_b128(rl, base2 + (unsigned)(((grp) * 4 + m) * 1024), 0, 0); } } while (0)
                LDQ(0, 0);
#pragma unroll
                for (int grp = 0; grp < 4; ++grp) {
                    const int a = grp >> 1, b = grp & 1, sl = grp & 1;
                    if (grp < 3) LDQ(sl ^ 1, grp + 1);
                    SCHED;
#pragma unroll
                    for (int m = 0; m < 4; ++m) {
                        const u32x4 c1 = q1[sl][m], c2 = q2[sl][m];
                        const f32x4 v0 = acc[a][b][m][0] + (f32x4){bf_lo(c1.x), bf_hi(c1.x), bf_lo(c1.y), bf_hi(c1.y)} + (f32x4){bf_lo(c2.x), bf_hi(c2.x), bf_lo(c2.y), bf_hi(c2.y)};
                        const f32x4 v1 = acc[a][b][m][1] + (f32x4){bf_lo(c1.z), bf_hi(c1.z), bf_lo(c1.w), bf_hi(c1.w)} + (f32x4){bf_lo(c2.z), bf_hi(c2.z), bf_lo(c2.w), bf_hi(c2.w)};
                        const unsigned ax = cvt_pk_bf16(silu(v0[0]), silu(v0[1])), ay = cvt_pk_bf16(silu(v0[2]), silu(v0[3]));
                        const unsigned bx = cvt_pk_bf16(silu(v1[0]), silu(v1[1])), by = cvt_pk_bf16(silu(v1[2]), silu(v1[3]));
                        const auto sx = __builtin_amdgcn_permlane16_swap(ax, bx, false, false);
                        const auto sy = __builtin_amdgcn_permlane16_swap(ay, by, false, false);
                        u32x4 o; o.x = sx[0]; o.y = sy[0]; o.z = sx[1]; o.w = sy[1];
                        if (bad) o = (u32x4){0x7fc07fc0u, 0x7fc07fc0u, 0x7fc07fc0u, 0x7fc07fc0u};
                        __builtin_amdgcn_raw_buffer_store_b128(o, ro, obase + (unsigned)(a * 128 + m * 16) * WROW + (unsigned)(b * 256), 0, 0);
                    }
                    SCHED;
                }
                BAR;
            }
            STAMP(5 + jn * 6);
        }
    }
    __syncthreads();
    if (threadIdx.x == 0) __hip_atomic_store((gu32*)(ws + WS_CALL + bid * 4), call, __ATOMIC_RELAXED, __HIP_MEMORY_SCOPE_AGENT);
    STAMP(13);
}

#define EARLY_COLS 12
void build_schedule(Params& p) {
    int order[32], n = 0;
    { const int head[8] = {0, 16, 17, 1, 18, 2, 19, 3}; for (int i = 0; i < 8; ++i) order[n++] = head[i]; }
    for (int c = 4; c < EARLY_COLS; ++c) order[n++] = 16 + c;
    for (int r = 4; r < 12; ++r) order[n++] = r;
    for (int c = EARLY_COLS; c < 16; ++c) order[n++] = 16 + c;
    for (int r = 12; r < 16; ++r) order[n++] = r;
    int when[32]; for (int i = 0; i < 32; ++i) when[order[i]] = i;
    for (int d = 0; d < 8; ++d) {
        int k = 0;
        for (int b = 0; b < 32; ++b) p.pos[d][b] = 0;
        if (d >= 2)
            for (int i = 0; i < 32; ++i) {
                const int b = order[i];
                if (d >= 4 && b >= 4 && b < 12 && ((b - 4) & 3) == d - 4) continue;
                for (int h = 0; h < 2; ++h) p.item[d][k++] = (unsigned char)((b >= 16 ? 64 : 0) | ((b & 15) << 1) | h);
                p.pos[d][b] = (unsigned char)k;
            }
        p.ninput[d] = (unsigned char)k;
        if (d == 1 || d >= 4) {
            int q = 0;
            for (int tm = 0; tm < 32; ++tm)
                for (int lr = 0; lr < 12; ++lr) {
                    if (d == 1 ? lr >= 4 : (lr < 4 || ((lr - 4) & 3) != d - 4)) continue;
                    for (int lc = 0; lc < 16; ++lc) {
                        const int t = when[lr] > when[16 + lc] ? when[lr] : when[16 + lc];
                        if (t != tm) continue;
                        p.item[d][k++] = (unsigned char)(lr * 16 + lc);
                        p.tpos[lr * 16 + lc] = (unsigned char)q++;
                    }
                }
        }
        p.nitems[d] = (unsigned char)k;
        for (; k < MAXITEMS; ++k) p.item[d][k] = 0;
    }
    for (int b = 0; b < 256; ++b) { p.job[b][0] = 0; p.job[b][1] = 0; }
    for (int x = 0; x < 8; ++x) {
        const int q = x >> 1, hx = x & 1;
        int j = 4;
        auto J = [](int type, int lr, int lc) { return (unsigned short)(type << 8 | lr << 4 | lc); };
        for (int i = 0; i < 4; ++i)
            for (int u = 0; u < 2; ++u) {
                const int b = (j++) * 8 + x;
                p.job[b][0] = J(1, 2 * u + hx, 4 * i + q);
                if (i < 2) p.job[b][1] = J(3, 12 + 2 + hx, 4 * (2 * i + u) + q);
            }
        for (int i = 0; i < 4; ++i)
            for (int u = 0; u < 4; ++u) {
                const int b = (j++) * 8 + x;
                p.job[b][0] = J(2, 4 + 2 * u + hx, 4 * i + q);
                if (i < 2) p.job[b][1] = J(4, 2 * hx + (u >> 1), 4 * (2 * (u & 1) + i) + q);
            }
        for (int i = 0; i < 4; ++i) {
            const int b = (j++) * 8 + x;
            p.job[b][0] = J(3, 12 + hx, 4 * i + q);
        }
    }
}

}

extern "C" void kernel_launch(int rank, int nranks, ncclComm_t comm,
                              hipStream_t stream,
                              void* const* d_in, const int* in_sizes, int n_in,
                              void* d_out, int out_size,
                              const int* in_dtypes, int out_dtype,
                              void* const* d_out_all,
                              void* const* d_ws_all, size_t ws_bytes) {
    (void)comm; (void)in_dtypes; (void)out_dtype; (void)d_out_all;
    if (nranks != NRANK || (unsigned)rank >= (unsigned)NRANK || n_in < 2 || ws_bytes < (size_t)WS_NEED ||
        in_sizes[0] != 8192 * 1024 || in_sizes[1] != 1024 * 8192 || out_size != 1024 * 8192) {
        fprintf(stderr, "kernel_launch: unexpected problem shape\n");
        return;
    }
    HIP_CHECK(hipFuncSetAttribute((const void*)gemm_rs_kernel, hipFuncAttributeMaxDynamicSharedMemorySize, LDS_BYTES));
    Params p;
    p.x = (const char*)d_in[0]; p.w = (const char*)d_in[1]; p.out = (char*)d_out;
    for (int i = 0; i < NRANK; ++i) p.ws[i] = (char*)d_ws_all[i];
    p.ws_bytes = (unsigned)WS_NEED; p.rank = rank;
    build_schedule(p);
    hipLaunchKernelGGL(gemm_rs_kernel, dim3(256), dim3(512), LDS_BYTES, stream, p);
    HIP_CHECK(hipGetLastError());
}
```

```cpp
#include <hip/hip_runtime.h>
#include <rccl/rccl.h>
#include <cstdio>
#include <cstdlib>

#define HIP_CHECK(expr) do { hipError_t _e = (expr); if (_e != hipSuccess) { \
    fprintf(stderr, "HIP error %s at %s:%d\n", hipGetErrorString(_e), __FILE__, __LINE__); \
    return; } } while (0)

namespace {

#define LAS __attribute__((address_space(3)))
#define GAS __attribute__((address_space(1)))
typedef short bf16x8 __attribute__((ext_vector_type(8)));
typedef short bf16x4 __attribute__((ext_vector_type(4)));
typedef float f32x4 __attribute__((ext_vector_type(4)));
typedef unsigned u32x4 __attribute__((ext_vector_type(4)));
typedef GAS unsigned gu32;

constexpr int NRANK = 8;
constexpr unsigned MiB = 1u << 20;
constexpr unsigned XROW = 2048;
constexpr unsigned WROW = 16384;
constexpr unsigned IN_BYTES = 16 * MiB;
constexpr unsigned OUT_BYTES = 16 * MiB;
constexpr unsigned TILE_BYTES = 128 * 1024;
constexpr unsigned WS_CALL = 0;
constexpr unsigned WS_TMO = 4096;
constexpr unsigned WS_TICK = 8192;
constexpr unsigned WS_FLAG = 16384;
constexpr unsigned ID_IN = 0;
constexpr unsigned ID_RP = 384;
constexpr unsigned ID_LP = 576;
constexpr unsigned WS_XR = 1 * MiB;
constexpr unsigned XARR = 8 * MiB, XR_PAR = 24 * MiB;
constexpr unsigned WS_WR = WS_XR + 48 * MiB;
constexpr unsigned WS_LP = WS_WR + 48 * MiB;
constexpr unsigned WS_RP = WS_LP + 32 * MiB;
constexpr unsigned RP_PAR = 24 * MiB;
constexpr unsigned WS_END = WS_RP + 48 * MiB;
constexpr unsigned WS_DBG = WS_END;
constexpr unsigned WS_NEED = WS_END + 1 * MiB;

constexpr int HTB = 16384;
constexpr int LDS_BYTES = 8 * HTB;
constexpr unsigned long long DEADLINE_TICKS = 50000000ull;
constexpr int MAXITEMS = 96, MAXJOBS = 2;

struct Params {
    const char* x; const char* w; char* out;
    char* ws[NRANK];
    int rank;
    int mode;
    int pace, slack, phase;
    unsigned char nitems[8], ninput[8];
    unsigned char item[8][MAXITEMS];
    unsigned char clink[32], cidx[32], cnum[8];
    unsigned char when[32], early[32];
    unsigned job[224][MAXJOBS];
};

__device__ __forceinline__ unsigned cvt_pk_bf16(float lo, float hi) {
    unsigned r; asm("v_cvt_pk_bf16_f32 %0, %1, %2" : "=v"(r) : "v"(lo), "v"(hi)); return r;
}
__device__ __forceinline__ float bf_lo(unsigned u) { return __uint_as_float(u << 16); }
__device__ __forceinline__ float bf_hi(unsigned u) { return __uint_as_float(u & 0xffff0000u); }
__device__ __forceinline__ float silu(float y) {
    const float c = fminf(fmaxf(y, -60.0f), 60.0f);
    return y * __builtin_amdgcn_rcpf(1.0f + __expf(-c));
}
__device__ __forceinline__ unsigned ld_sys(const char* p) {
    return __hip_atomic_load((const gu32*)p, __ATOMIC_RELAXED, __HIP_MEMORY_SCOPE_SYSTEM);
}
__device__ __forceinline__ unsigned ld_agent(const char* p) {
    return __hip_atomic_load((const gu32*)p, __ATOMIC_RELAXED, __HIP_MEMORY_SCOPE_AGENT);
}
__device__ __forceinline__ unsigned grow_of(int g, int lr) {
    if (lr < 4) return (unsigned)((g ^ 1) * 1024 + lr * 256);
    if (lr >= 12) return (unsigned)(g * 1024 + (lr - 12) * 256);
    const int r = lr - 4;
    return (unsigned)((g ^ (4 + (r & 3))) * 1024 + ((g >> 1) & 1) * 512 + (r >> 2) * 256);
}
__device__ __forceinline__ bool give_up(char* ws, unsigned call, unsigned long long t0) {
    const unsigned t = __builtin_amdgcn_readfirstlane(ld_agent(ws + WS_TMO));
    if (t == call) return true;
    if (__builtin_amdgcn_s_memrealtime() - t0 > DEADLINE_TICKS) {
        __hip_atomic_store((gu32*)(ws + WS_TMO), call, __ATOMIC_RELAXED, __HIP_MEMORY_SCOPE_AGENT);
        return true;
    }
    return false;
}

#define STAMP(i) do {} while (0)

#define WAIT_V(n) asm volatile("s_waitcnt vmcnt(" #n ")" ::: "memory")
#define WAIT_L(n) asm volatile("s_waitcnt lgkmcnt(" #n ")" ::: "memory")
#define BAR asm volatile("s_barrier" ::: "memory")
#define SCHED __builtin_amdgcn_sched_barrier(0)
#define RFL(x) __builtin_amdgcn_readfirstlane(x)

#define CWAVES 1
#define CWIN 36
constexpr unsigned OOB = 0x40000000u;
__device__ __forceinline__ void comm_role(const Params& p, const int cid, const unsigned call) {
    const int g = p.rank, d = p.clink[cid] & 7, peer = g ^ d, cix = p.cidx[cid], cn = p.cnum[d];
    const int wid = RFL(threadIdx.x >> 6);
    if (d == 0 || (unsigned)peer >= (unsigned)NRANK || wid >= CWAVES) return;
    char* ws = p.ws[g];
    const int lane = threadIdx.x & 63;
    const unsigned par = call & 1u;
    const int nit = p.nitems[d], nin = p.ninput[d], nun = nit * 4;
    const unsigned ps3 = peer & 1, ps1 = (peer >> 2) & 1, slot = (unsigned)(d - 2);
    gu32* tick = (gu32*)(ws + WS_TICK + d * 256 + par * 128u);
    if (cix == 0 && threadIdx.x == 0)
        __hip_atomic_store((gu32*)(ws + WS_TICK + d * 256 + (par ^ 1u) * 128u), 0u, __ATOMIC_RELAXED, __HIP_MEMORY_SCOPE_AGENT);
    unsigned dsrc[2], ddst[2], dfg[2];
#pragma unroll
    for (int h = 0; h < 2; ++h) {
        const int i = h * 64 + lane;
        const unsigned it = p.item[d][i < MAXITEMS ? i : 0];
        const unsigned blk = (it >> 1) & 15u, hh = it & 1u;
        dfg[h] = ID_IN + slot * 64u + it;
        if (i >= nin) {
            const unsigned lr = it >> 4, lc = it & 15u;
            const unsigned dt = lr < 4u ? it : 64u + (unsigned)(g & 1) * 64u + ((unsigned)((g >> 1) & 1) * 2u + ((lr - 4u) >> 2)) * 16u + lc;
            dsrc[h] = WS_LP + it * TILE_BYTES;
            ddst[h] = (WS_RP + par * RP_PAR + dt * TILE_BYTES) | 2u;
            dfg[h] = (ID_RP + dt) | (ID_LP + it) << 16;
        } else if (it & 32u) {
            dsrc[h] = (ps3 * 512u + hh * 256u) * WROW + ps1 * 8192u + blk * 512u;
            ddst[h] = (WS_WR + (slot * 512u + hh * 256u) * WROW + par * 8192u + blk * 512u) | 1u;
        } else {
            dsrc[h] = (grow_of(peer, (int)blk) + hh * 128u) * XROW + ps3 * 1024u;
            ddst[h] = WS_XR + par * XR_PAR + (slot >> 1) * XARR + (blk * 256u + hh * 128u) * XROW + (slot & 1u) * 1024u;
        }
    }
    char* pb = p.ws[peer];
    const u32x4 rd = {(unsigned)(size_t)pb, (unsigned)((size_t)pb >> 32), WS_END, 0x00020000u};
    const unsigned lane_lin = (unsigned)lane * 16u, lane_w = (unsigned)(lane >> 5) * WROW + (unsigned)(lane & 31) * 16u;
    gu32* flags = (gu32*)(pb + WS_FLAG);
    const u32x4 rl = {(unsigned)(size_t)ws, (unsigned)((size_t)ws >> 32), WS_XR, 0x00020000u};
    const unsigned long long t0 = __builtin_amdgcn_s_memrealtime();
    const int nw = cn * CWAVES, pace = p.pace, slack = p.slack;
    int ub = cix * CWAVES + wid;
    unsigned long long rmask = 0;
    u32x4 X[4][8], R = {0u, 0u, 0u, 0u};
#pragma unroll
    for (int s = 0; s < 4; ++s)
#pragma unroll
        for (int u = 0; u < 8; ++u) X[s][u] = (u32x4){0u, 0u, 0u, 0u};
    unsigned T = 0, voa = lane_lin, dofa = OOB, stra = 0;
    int fa = -1, fprev = -1, duea = 0, tbase = 0;
    bool seen = false;
    const unsigned one = 1u;
    constexpr int LAG = (CWIN + 14) / 16;
    static_assert(CWIN <= 48 && LAG <= 3, "window");
#define MV1(k) "s_waitcnt vmcnt(%16)\n\tbuffer_store_dwordx4 %" #k ", %10, %11, %8 offen sc0 sc1\n\ts_add_u32 %8, %8, %12\n\t" \
        "buffer_load_dwordx4 %" #k ", %13, %14, %9 offen sc1\n\ts_add_u32 %9, %9, %15\n\t"
#define PUBLISH(fw) do { if (lane == 0) __hip_atomic_store(flags + ((fw) & 0xffff), call, __ATOMIC_RELAXED, __HIP_MEMORY_SCOPE_SYSTEM); CSTAMP(fw); } while (0)
#define CSTAMP(fw) do {} while (0)
#define GATES(go, itn) const int _c = ((lane - (itn)) & 63) + (itn); const unsigned _gid = (_c < 64 ? dfg[0] : dfg[1]) >> 16; \
        const bool look = _c < (itn) + 8 && _c < nit && _gid != 0u; const unsigned go = look ? WS_FLAG + _gid * 16u : OOB
#define OPEN(r) __ballot((r).x >= call && (r).y >= call && (r).z >= call && (r).w >= call)
    for (;;) {
        const int itn = ub >> 2;
        const bool live = ub < nun;
        if (!live && fa < 0) break;
        const unsigned fg = (unsigned)(itn < 64 ? __builtin_amdgcn_readlane((int)dfg[0], itn & 63) : __builtin_amdgcn_readlane((int)dfg[1], itn & 63));
        if (seen) rmask = OPEN(R);
        bool real = live && ((fg >> 16) == 0u || ((rmask >> (itn & 63)) & 1ull));
        GATES(go, itn);
        if (live && !real && fa < 0) {
            WAIT_V(0);
            if (fprev >= 0) { PUBLISH(fprev); fprev = -1; }
            for (unsigned polls = 1;; ++polls) {
                asm volatile("buffer_load_dwordx4 %0, %1, %2, 0 offen sc0 sc1\n\ts_waitcnt vmcnt(0)" : "+v"(R) : "v"(go), "s"(rl) : "memory");
                rmask = OPEN(R);
                if ((rmask >> (itn & 63)) & 1ull) break;
                if ((polls & 63u) == 0u && give_up(ws, call, t0)) return;
                __builtin_amdgcn_s_sleep(2);
            }
            real = true;
        }
        const unsigned so = (unsigned)(itn < 64 ? __builtin_amdgcn_readlane((int)dsrc[0], itn & 63) : __builtin_amdgcn_readlane((int)dsrc[1], itn & 63));
        const unsigned dk = (unsigned)(itn < 64 ? __builtin_amdgcn_readlane((int)ddst[0], itn & 63) : __builtin_amdgcn_readlane((int)ddst[1], itn & 63));
        const unsigned kind = dk & 3u;
        const unsigned strb = RFL(kind == 0u ? XROW : (kind == 1u ? 2u * WROW : 1024u));
        const char* sb = kind == 0u ? p.x : (kind == 1u ? p.w : ws);
        const u32x4 rs = {(unsigned)(size_t)sb, (unsigned)((size_t)sb >> 32), sb == ws ? WS_END : IN_BYTES, 0x00020000u};
        const unsigned uofs = (unsigned)((ub & 3) * 32) * strb;
        const unsigned vob = kind == 1u ? lane_w : lane_lin;
        unsigned sof = RFL(real ? so + uofs : OOB), dof = RFL(dofa);
        const unsigned sta = RFL(stra);
        if (real && lane == 0) asm volatile("global_atomic_add %0, %1, %2, off sc0" : "+v"(T) : "v"(tick), "v"(one) : "memory");
        seen = false;
#pragma unroll
        for (int s = 0; s < 4; ++s) {
            if (fa >= 0) {
                int lead;
                while ((lead = tbase + duea + ((s * nw * pace) >> 2) - (int)(unsigned)(__builtin_amdgcn_s_memrealtime() - t0) * 16) > 0) { }
                if (lead < -slack) tbase -= lead + slack;
            }
            if (s == LAG && fprev >= 0) { asm volatile("s_waitcnt vmcnt(%0)" :: "i"(CWIN) : "memory"); PUBLISH(fprev); }
            asm volatile("s_nop 4\n\t" MV1(0) MV1(1) MV1(2) MV1(3) MV1(4) MV1(5) MV1(6) MV1(7)
                : "+v"(X[s][0]), "+v"(X[s][1]), "+v"(X[s][2]), "+v"(X[s][3]), "+v"(X[s][4]), "+v"(X[s][5]), "+v"(X[s][6]), "+v"(X[s][7]), "+s"(dof), "+s"(sof)
                : "v"(voa), "s"(rd), "s"(sta), "v"(vob), "s"(rs), "s"(strb), "i"(CWIN) : "memory", "scc");
            if (s == 0 && live && __any(look)) {
                asm volatile("buffer_load_dwordx4 %0, %1, %2, 0 offen sc0 sc1" : "+v"(R) : "v"(go), "s"(rl) : "memory");
                seen = true;
            }
        }
        asm volatile("s_waitcnt vmcnt(48)" : "+v"(R), "+v"(T) :: "memory");
        fprev = fa;
        fa = real ? (int)((fg & 0xffffu) * 4u + (unsigned)(ub & 3)) | itn << 16 : -1;
        voa = vob; stra = strb; dofa = real ? (dk & ~3u) + uofs : OOB;
        duea = ub * pace + (((ub & 3) * p.phase) >> 2);
        if (real) ub = RFL((int)T) + nw;
    }
    WAIT_V(0);
    if (fprev >= 0) PUBLISH(fprev);
}

__global__ __launch_bounds__(512, 2) void gemm_rs_kernel(const Params p) {
    extern __shared__ __attribute__((aligned(16))) unsigned char shm[];
    LAS unsigned char* lds = (LAS unsigned char*)shm;
    const int g = p.rank;
    char* ws = p.ws[g];
    const int bid = blockIdx.x;
    const unsigned call = __builtin_amdgcn_readfirstlane(ld_agent(ws + WS_CALL + bid * 4)) + 1u;
    STAMP(0);
    if (bid < 32) {
        comm_role(p, bid, call);
    } else {
        const int tid = threadIdx.x, wid = __builtin_amdgcn_readfirstlane(tid >> 6), lane = tid & 63;
        const int wr = wid >> 2, wc = wid & 3, fr = lane & 15, fq = lane >> 4;
        const unsigned par = call & 1u;
        const int s1 = (g >> 2) & 1;
        unsigned voffA, voffB;
        { const int r = wid * 8 + (lane >> 3); voffA = (unsigned)r * XROW + (unsigned)(((lane & 7) ^ ((r >> 1) & 7)) * 16); }
        { const int r = wid * 4 + (lane >> 4); voffB = (unsigned)r * WROW + (unsigned)(((lane & 15) ^ (((r & 3) << 2) | (((r >> 3) & 1) << 1))) * 16); }
        const unsigned ldsw = (unsigned)(size_t)lds + (unsigned)wid * 1024u;
        const int aoff0 = (wr * 64 + fr) * 128 + ((fq ^ (fr >> 1)) * 16), aoff1 = aoff0 ^ 64;
        int boff0, boff1;
        { const int q = (lane & 15) >> 2, pp = lane & 3, r = 8 * fq + q;
          boff0 = 4 * HTB + r * 256 + (((wc * 4 + (pp >> 1)) ^ ((q << 2) | ((fq & 1) << 1))) * 16) + (pp & 1) * 8; boff1 = boff0 ^ 32; }
        asm volatile("" : "+v"(boff0), "+v"(boff1));
#define SA(b, h) (((b) * 2 + (h)) * HTB)
#define SB(b, h) (((b) * 2 + (h)) * HTB)
#define DMA2(rs, voff, so, l0, o0, o1) do { unsigned _t; asm volatile( \
        "s_add_u32 m0, %3, %5\n\ts_add_u32 %0, %4, %7\n\tbuffer_load_dwordx4 %1, %2, %0 offen lds\n\t" \
        "s_add_u32 m0, %3, %6\n\ts_add_u32 %0, %4, %8\n\tbuffer_load_dwordx4 %1, %2, %0 offen lds" \
        : "=&s"(_t) : "v"(voff), "s"(rs), "s"(ldsw), "s"(so), "i"(l0), "i"((l0) + 8192), "i"(o0), "i"(o1) : "memory", "scc"); } while (0)
#define STAGE_A(b, h, so, odd) DMA2(ra, voffA, so, SA(b, h), (h) * 128 * XROW + (odd) * 128, ((h) * 128 + 64) * XROW + (odd) * 128)
#define STAGE_B(b, h, so, odd) DMA2(rb, voffB, so, 4 * HTB + SB(b, h), (h) * 256 + (odd) * 64 * WROW, (h) * 256 + ((odd) * 64 + 32) * WROW)
#define LDA(dst, b, h) do { _Pragma("unroll") for (int m = 0; m < 4; ++m) { \
        dst[m][0] = *(const LAS bf16x8*)(lds + SA(b, h) + aoff0 + m * 2048); dst[m][1] = *(const LAS bf16x8*)(lds + SA(b, h) + aoff1 + m * 2048); } } while (0)
#define TR(off) __builtin_amdgcn_ds_read_tr16_b64_v4i16((LAS bf16x4*)(lds + (off)))
#define LDB(dst, b, h) do { _Pragma("unroll") for (int k = 0; k < 2; ++k) { \
        dst[0][k] = __builtin_shufflevector(TR(SB(b, h) + boff0 + k * 8192), TR(SB(b, h) + boff0 + k * 8192 + 1024), 0, 1, 2, 3, 4, 5, 6, 7); \
        dst[1][k] = __builtin_shufflevector(TR(SB(b, h) + boff1 + k * 8192), TR(SB(b, h) + boff1 + k * 8192 + 1024), 0, 1, 2, 3, 4, 5, 6, 7); } } while (0)
#define PIN(ai, bj) asm volatile("" : "+v"(acc[ai][bj][0][0]), "+v"(acc[ai][bj][0][1]), "+v"(acc[ai][bj][1][0]), "+v"(acc[ai][bj][1][1]), \
        "+v"(acc[ai][bj][2][0]), "+v"(acc[ai][bj][2][1]), "+v"(acc[ai][bj][3][0]), "+v"(acc[ai][bj][3][1]))
#define MMA(ai, bj, At, Bt) do { __builtin_amdgcn_s_setprio(1); PIN(ai, bj); \
        _Pragma("unroll") for (int m = 0; m < 4; ++m) _Pragma("unroll") for (int n = 0; n < 2; ++n) _Pragma("unroll") for (int k = 0; k < 2; ++k) \
        acc[ai][bj][m][n] = __builtin_amdgcn_mfma_f32_16x16x32_bf16(Bt[n][k], At[m][k], acc[ai][bj][m][n], 0, 0, 0); \
        PIN(ai, bj); __builtin_amdgcn_s_setprio(0); } while (0)

        for (int jn = 0; jn < MAXJOBS; ++jn) {
            const unsigned job = p.job[bid - 32][jn];
            const int type = (job >> 8) & 7, lr = (job >> 4) & 15, lc = job & 15;
            if (type == 0) break;
            f32x4 acc[2][2][4][2];
#pragma unroll
            for (int a = 0; a < 2; ++a)
#pragma unroll
                for (int b = 0; b < 2; ++b)
#pragma unroll
                    for (int m = 0; m < 4; ++m)
#pragma unroll
                        for (int n = 0; n < 2; ++n) acc[a][b][m][n] = (f32x4){0.f, 0.f, 0.f, 0.f};
            bool late = false;
            for (int ph = type == 2 ? 1 : 0; ph < 3; ++ph) {
                const char* abase; const char* bbase; unsigned a0, b0, nbytes, segs; int nt;
                if (ph == 0) {
                    const unsigned grow = type == 4 ? (unsigned)(g * 1024 + lr * 256) : grow_of(g, lr);
                    const unsigned gcol = (unsigned)((type == 4 ? 1 - s1 : s1) * 4096 + lc * 256);
                    abase = p.x; bbase = p.w; nbytes = IN_BYTES;
                    a0 = grow * XROW; b0 = gcol * 2u; nt = 16; segs = 0x10u;
                } else {
                    const unsigned mask = (job >> (5 + 6 * ph)) & 63u;
                    if (mask == 0u) break;
                    abase = ws; bbase = ws; nbytes = WS_END;
                    a0 = WS_XR + par * XR_PAR + (unsigned)lr * 256u * XROW; b0 = WS_WR + par * 8192u + (unsigned)lc * 512u;
                    segs = 0u; nt = 0;
                    for (int sl = 0; sl < 6; ++sl) if ((mask >> sl) & 1u) { segs |= (unsigned)sl << (nt >> 1); nt += 8; }
                    if (ph == 1) STAMP(1 + jn * 6);
                    if (wid == 0) {
                        const unsigned sl = ((mask >> (lane >> 3)) & 1u) ? (unsigned)(lane >> 3) : (segs & 15u);
                        const char* fx = ws + WS_FLAG + (ID_IN + sl * 64u + (unsigned)lr * 2u) * 16u + (lane & 7) * 4;
                        const char* fw = ws + WS_FLAG + (ID_IN + sl * 64u + (unsigned)(16 + lc) * 2u) * 16u + (lane & 7) * 4;
                        const unsigned long long t0 = __builtin_amdgcn_s_memrealtime();
                        {
                            const unsigned eb = p.early[p.when[lr] > p.when[16 + lc] ? lr : 16 + lc];
                            if (eb != 255u) {
                                const char* fe = ws + WS_FLAG + (ID_IN + eb * 2u + 1u) * 16u;
                                for (unsigned polls = 1; RFL(ld_sys(fe)) < call; ++polls) {
                                    if ((polls & 15u) == 0u && give_up(ws, call, t0)) break;
                                    __builtin_amdgcn_s_sleep(100);
                                }
                            }
                        }
                        for (unsigned polls = 1;; ++polls) {
                            const unsigned f0 = ld_sys(fx), f1 = ld_sys(fw);
                            if (__all(f0 >= call && f1 >= call)) break;
                            if ((polls & 15u) == 0u && give_up(ws, call, t0)) { late = true; break; }
                            __builtin_amdgcn_s_sleep(8);
                        }
                        __builtin_amdgcn_fence(__ATOMIC_ACQUIRE, "");
                        WAIT_V(0);
                    }
                    asm volatile("" ::: "memory"); BAR; asm volatile("" ::: "memory");
                    if (ph == 1) STAMP(2 + jn * 6);
                }
                bf16x8 At[4][2], B0[2][2], B1[2][2];
                const u32x4 ra = {(unsigned)(size_t)abase, (unsigned)((size_t)abase >> 32), nbytes, 0x00020000u};
                const u32x4 rb = {(unsigned)(size_t)bbase, (unsigned)((size_t)bbase >> 32), nbytes, 0x00020000u};
#define KOFF(t, oa, ob) do { const int _s = (segs >> (((t) >> 3) * 4)) & 15u; \
                oa = a0 + (unsigned)(_s >> 1) * XARR + (unsigned)(_s & 1) * 1024u + (unsigned)((t) & 7) * 128u; \
                ob = b0 + (unsigned)(_s * 8 + ((t) & 7)) * (64u * WROW); } while (0)
                unsigned ca, cb;
                KOFF(0, ca, cb);
                STAGE_B(0, 0, cb, 0); STAGE_A(0, 0, ca, 0); STAGE_B(0, 1, cb, 0); STAGE_A(0, 1, ca, 0);
                if (wr == 1) BAR;
                WAIT_V(4); BAR;
                STAGE_B(1, 0, cb, 1); STAGE_A(1, 0, ca, 1); STAGE_B(1, 1, cb, 1);
                WAIT_V(6); BAR;
                for (int t = 0; t < nt; t += 2) {
                    const int tn = t + 2 < nt ? t + 2 : 0;
                    unsigned a2, b2;
                    KOFF(tn, a2, b2);
                    LDB(B0, 0, 0); SCHED; LDA(At, 0, 0); STAGE_A(1, 1, ca, 1);
                    WAIT_L(8); BAR; WAIT_L(0); MMA(0, 0, At, B0); BAR; SCHED;
                    LDB(B1, 0, 1); STAGE_B(0, 0, b2, 0);
                    BAR; WAIT_L(0); MMA(0, 1, At, B1); BAR;
                    LDA(At, 0, 1); STAGE_A(0, 0, a2, 0);
                    BAR; WAIT_L(0); MMA(1, 0, At, B0); BAR; SCHED;
                    STAGE_B(0, 1, b2, 0);
                    WAIT_V(6); BAR; MMA(1, 1, At, B1); BAR;
                    LDB(B0, 1, 0); SCHED; LDA(At, 1, 0); STAGE_A(0, 1, a2, 0);
                    WAIT_L(8); BAR; WAIT_L(0); MMA(0, 0, At, B0); BAR; SCHED;
                    LDB(B1, 1, 1); STAGE_B(1, 0, b2, 1);
                    BAR; WAIT_L(0); MMA(0, 1, At, B1); BAR;
                    LDA(At, 1, 1); STAGE_A(1, 0, a2, 1);
                    BAR; WAIT_L(0); MMA(1, 0, At, B0); BAR; SCHED;
                    STAGE_B(1, 1, b2, 1);
                    WAIT_V(6); BAR; MMA(1, 1, At, B1); BAR;
                    ca = a2;
                }
                WAIT_V(0);
                if (wr == 0) BAR;
                BAR;
            }
            STAMP(3 + jn * 6);
            int tz = threadIdx.x; asm volatile("" : "+v"(tz));
            const int elane = tz & 63;
            const unsigned toff = (unsigned)wid * 16384u + (unsigned)elane * 16u;
            const auto rl = __builtin_amdgcn_make_buffer_rsrc((void*)ws, 0, WS_END, 0x00020000);
            const bool helped = (job >> 29) & 1u, fin = type >= 3;
            const unsigned tl = type == 2 ? 192u + ((job >> 23) & 63u) : (unsigned)(lr * 16 + lc);
            const unsigned pbase = WS_LP + tl * TILE_BYTES + toff;
            if (!fin && !helped) {
#pragma unroll
                for (int a = 0; a < 2; ++a)
#pragma unroll
                    for (int b = 0; b < 2; ++b)
#pragma unroll
                        for (int m = 0; m < 4; ++m) {
                            const f32x4 v0 = acc[a][b][m][0], v1 = acc[a][b][m][1];
                            u32x4 o; o.x = cvt_pk_bf16(v0[0], v0[1]); o.y = cvt_pk_bf16(v0[2], v0[3]); o.z = cvt_pk_bf16(v1[0], v1[1]); o.w = cvt_pk_bf16(v1[2], v1[3]);
                            __builtin_amdgcn_raw_buffer_store_b128(o, rl, pbase + (unsigned)(((a * 2 + b) * 4 + m) * 1024), 0, 16);
                        }
            } else {
                const int orow = type == 3 ? lr - 12 : lr;
                const unsigned t1 = type == 3 ? (unsigned)(orow * 16 + lc) : 64u + (unsigned)(orow * 16 + lc);
                const unsigned ht = 192u + ((job >> 23) & 63u);
                LAS unsigned* lflag = (LAS unsigned*)lds;
                if (wid == 0) {
                    const unsigned id = !fin ? ID_LP + ht : ID_RP + t1 + (type == 4 && elane >= 32 ? 64u : 0u);
                    const char* fp = ws + WS_FLAG + id * 16u + (elane & 3) * 4;
                    const unsigned long long t0 = __builtin_amdgcn_s_memrealtime();
                    for (unsigned polls = 1;; ++polls) {
                        const unsigned f = ld_sys(fp);
                        if (__all(f >= call)) break;
                        if ((polls & 15u) == 0u && give_up(ws, call, t0)) { late = true; break; }
                        __builtin_amdgcn_s_sleep(6);
                    }
                    __builtin_amdgcn_fence(__ATOMIC_ACQUIRE, "");
                    const unsigned tm = ld_agent(ws + WS_TMO);
                    if (elane == 0) lflag[0] = tm == call ? 1u : 0u;
                    asm volatile("s_waitcnt vmcnt(0) lgkmcnt(0)" ::: "memory");
                }
                BAR;
                STAMP(4 + jn * 6);
                const bool bad = RFL(lflag[0]) != 0u;
                const unsigned base1 = bad ? WS_END : (fin ? WS_RP + par * RP_PAR + t1 * TILE_BYTES : WS_LP + ht * TILE_BYTES) + toff;
                const unsigned base2 = (bad || type != 4) ? WS_END : base1 + 64u * TILE_BYTES;
                const int efr = elane & 15, efq = elane >> 4;
                const auto ro = __builtin_amdgcn_make_buffer_rsrc((void*)p.out, 0, OUT_BYTES, 0x00020000);
                const unsigned ocol = (unsigned)((type == 3 ? s1 : 1 - s1) * 4096 + lc * 256 + wc * 32 + (efq & 1) * 16 + (efq >> 1) * 8);
                const unsigned obase = (unsigned)(orow * 256 + wr * 64 + efr) * WROW + ocol * 2u;
                u32x4 q1[2][4], q2[2][4];
#define LDQ(s, grp) do { _Pragma("unroll") for (int m = 0; m < 4; ++m) { \
                    q1[s][m] = __builtin_amdgcn_raw_buffer_load_b128(rl, base1 + (unsigned)(((grp) * 4 + m) * 1024), 0, 0); \
                    q2[s][m] = __builtin_amdgcn_raw_buffer_load_b128(rl, base2 + (unsigned)(((grp) * 4 + m) * 1024), 0, 0); } } while (0)
                LDQ(0, 0);
#pragma unroll
                for (int grp = 0; grp < 4; ++grp) {
                    const int a = grp >> 1, b = grp & 1, sl = grp & 1;
                    if (grp < 3) LDQ(sl ^ 1, grp + 1);
                    SCHED;
#pragma unroll
                    for (int m = 0; m < 4; ++m) {
                        const u32x4 c1 = q1[sl][m], c2 = q2[sl][m];
                        const f32x4 v0 = acc[a][b][m][0] + (f32x4){bf_lo(c1.x), bf_hi(c1.x), bf_lo(c1.y), bf_hi(c1.y)} + (f32x4){bf_lo(c2.x), bf_hi(c2.x), bf_lo(c2.y), bf_hi(c2.y)};
                        const f32x4 v1 = acc[a][b][m][1] + (f32x4){bf_lo(c1.z), bf_hi(c1.z), bf_lo(c1.w), bf_hi(c1.w)} + (f32x4){bf_lo(c2.z), bf_hi(c2.z), bf_lo(c2.w), bf_hi(c2.w)};
                        if (fin) {
                            const unsigned ax = cvt_pk_bf16(silu(v0[0]), silu(v0[1])), ay = cvt_pk_bf16(silu(v0[2]), silu(v0[3]));
                            const unsigned bx = cvt_pk_bf16(silu(v1[0]), silu(v1[1])), by = cvt_pk_bf16(silu(v1[2]), silu(v1[3]));
                            const auto sx = __builtin_amdgcn_permlane16_swap(ax, bx, false, false);
                            const auto sy = __builtin_amdgcn_permlane16_swap(ay, by, false, false);
                            u32x4 o; o.x = sx[0]; o.y = sy[0]; o.z = sx[1]; o.w = sy[1];
                            if (bad) o = (u32x4){0x7fc07fc0u, 0x7fc07fc0u, 0x7fc07fc0u, 0x7fc07fc0u};
                            __builtin_amdgcn_raw_buffer_store_b128(o, ro, obase + (unsigned)(a * 128 + m * 16) * WROW + (unsigned)(b * 256), 0, 0);
                        } else {
                            u32x4 o; o.x = cvt_pk_bf16(v0[0], v0[1]); o.y = cvt_pk_bf16(v0[2], v0[3]); o.z = cvt_pk_bf16(v1[0], v1[1]); o.w = cvt_pk_bf16(v1[2], v1[3]);
                            __builtin_amdgcn_raw_buffer_store_b128(o, rl, pbase + (unsigned)((grp * 4 + m) * 1024), 0, 16);
                        }
                    }
                    SCHED;
                }
            }
            if (!fin) WAIT_V(0);
            BAR;
            if (!fin && tz < 4 && !late)
                __hip_atomic_store((gu32*)(ws + WS_FLAG + (ID_LP + tl) * 16u + tz * 4), call, __ATOMIC_RELAXED, __HIP_MEMORY_SCOPE_AGENT);
            STAMP(5 + jn * 6);
        }
    }
    __syncthreads();
    if (threadIdx.x == 0) __hip_atomic_store((gu32*)(ws + WS_CALL + bid * 4), call, __ATOMIC_RELAXED, __HIP_MEMORY_SCOPE_AGENT);
    STAMP(13);
}

#define CLINKS "01234567012345670123456701234567"
#define PACE 892
#define PHASE 0
#define SLACK 1800
#define SPLIT_COLS 8
void build_schedule(Params& p) {
    int order[32], n = 0;
    {
        const int dflt[32] = {0, 16, 17, 1, 18, 2, 19, 3, 20, 21, 22, 23, 24, 25, 4, 5, 6, 7, 26, 27, 28, 8, 9, 10, 11, 29, 30, 31, 12, 13, 14, 15};
        for (n = 0; n < 32; ++n) order[n] = dflt[n];
    }
    int nsplit = SPLIT_COLS;
    p.pace = PACE; p.slack = SLACK; p.phase = PHASE;
    {
        const char* cl = CLINKS;
        for (int d = 0; d < 8; ++d) p.cnum[d] = 0;
        for (int b = 0; b < 32; ++b) { const int d = (cl[b] - '0') & 7; p.clink[b] = (unsigned char)d; p.cidx[b] = p.cnum[d]++; }
    }
    int when[32];
    for (int i = 0; i < 32; ++i) { when[order[i]] = i; p.when[order[i]] = (unsigned char)i; p.early[order[i]] = (unsigned char)(i >= 2 ? order[i - 2] : 255); }
    for (int d = 0; d < 8; ++d) {
        int k = 0;
        if (d >= 2)
            for (int i = 0; i < 32; ++i) {
                const int b = order[i];
                if (d >= 4 && b >= 4 && b < 12 && ((b - 4) & 3) == d - 4) continue;
                for (int h = 0; h < 2; ++h) p.item[d][k++] = (unsigned char)(b << 1 | h);
            }
        p.ninput[d] = (unsigned char)k;
        if (d == 1 || d >= 4)
            for (int tm = 0; tm < 32; ++tm)
                for (int lr = 0; lr < 12; ++lr) {
                    if (d == 1 ? lr >= 4 : (lr < 4 || ((lr - 4) & 3) != d - 4)) continue;
                    for (int lc = 0; lc < 16; ++lc)
                        if ((when[lr] > when[16 + lc] ? when[lr] : when[16 + lc]) == tm) p.item[d][k++] = (unsigned char)(lr * 16 + lc);
                }
        p.nitems[d] = (unsigned char)k;
        for (; k < MAXITEMS; ++k) p.item[d][k] = 0;
    }
    for (int b = 0; b < 224; ++b) for (int j = 0; j < MAXJOBS; ++j) p.job[b][j] = 0;
    auto J = [](int type, int lr, int lc, int ma, int mb) { return (unsigned)(lc | lr << 4 | type << 8 | ma << 11 | mb << 17); };
    for (int x = 0; x < 8; ++x) {
        const int q = x >> 1, hx = x & 1;
        int j = 0;
        for (int i = 0; i < 4; ++i)
            for (int u = 0; u < 2; ++u) {
                const int b = (j++) * 8 + x, lc = 4 * i + q;
                p.job[b][0] = J(1, 2 * u + hx, lc, 63, 0);
                if (i < 2) {
                    const int hb = (24 + 2 * i + u) * 8 + x, ht = (2 * i + u) * 8 + x;
                    int n = 0;
                    if (lc < nsplit) {
                        p.job[b][0] = J(1, 2 * u + hx, lc, 7, 0) | (unsigned)ht << 23 | 1u << 29;
                        p.job[hb][n++] = J(2, 2 * u + hx, lc, 56, 0) | (unsigned)ht << 23;
                    }
                    p.job[hb][n] = J(3, 12 + hx, 4 * (2 * i + u) + q, 63, 0);
                    p.job[b][1] = J(3, 12 + 2 + hx, 4 * (2 * i + u) + q, 63, 0);
                }
            }
        for (int i = 0; i < 4; ++i)
            for (int u = 0; u < 4; ++u) {
                const int b = (j++) * 8 + x, r = 2 * u + hx;
                p.job[b][0] = J(1, 4 + r, 4 * i + q, 63 & ~(4 << (r & 3)), 0);
                if (i < 2) p.job[b][1] = J(4, 2 * hx + (u >> 1), 4 * (2 * (u & 1) + i) + q, 0, 0);
            }
    }
}

}

extern "C" void kernel_launch(int rank, int nranks, ncclComm_t comm,
                              hipStream_t stream,
                              void* const* d_in, const int* in_sizes, int n_in,
                              void* d_out, int out_size,
                              const int* in_dtypes, int out_dtype,
                              void* const* d_out_all,
                              void* const* d_ws_all, size_t ws_bytes) {
    (void)comm; (void)in_dtypes; (void)out_dtype; (void)d_out_all;
    if (nranks != NRANK || (unsigned)rank >= (unsigned)NRANK || n_in < 2 || ws_bytes < (size_t)WS_NEED ||
        in_sizes[0] != 8192 * 1024 || in_sizes[1] != 1024 * 8192 || out_size != 1024 * 8192) {
        fprintf(stderr, "kernel_launch: unexpected problem shape\n");
        return;
    }
    HIP_CHECK(hipFuncSetAttribute((const void*)gemm_rs_kernel, hipFuncAttributeMaxDynamicSharedMemorySize, LDS_BYTES));
    Params p;
    p.x = (const char*)d_in[0]; p.w = (const char*)d_in[1]; p.out = (char*)d_out;
    for (int i = 0; i < NRANK; ++i) p.ws[i] = (char*)d_ws_all[i];
    p.rank = rank;
    build_schedule(p);
    hipLaunchKernelGGL(gemm_rs_kernel, dim3(256), dim3(512), LDS_BYTES, stream, p);
    HIP_CHECK(hipGetLastError());
}
```

```cpp
#include <hip/hip_runtime.h>
#include <rccl/rccl.h>
#include <cstdio>
#include <cstdlib>

#define HIP_CHECK(expr) do { hipError_t _e = (expr); if (_e != hipSuccess) { \
    fprintf(stderr, "HIP error %s at %s:%d\n", hipGetErrorString(_e), __FILE__, __LINE__); \
    return; } } while (0)

namespace {

#define LAS __attribute__((address_space(3)))
#define GAS __attribute__((address_space(1)))
typedef short bf16x8 __attribute__((ext_vector_type(8)));
typedef short bf16x4 __attribute__((ext_vector_type(4)));
typedef float f32x4 __attribute__((ext_vector_type(4)));
typedef unsigned u32x4 __attribute__((ext_vector_type(4)));
typedef GAS unsigned gu32;

constexpr int NRANK = 8;
constexpr unsigned MiB = 1u << 20;
constexpr unsigned XROW = 2048;
constexpr unsigned WROW = 16384;
constexpr unsigned IN_BYTES = 16 * MiB;
constexpr unsigned OUT_BYTES = 16 * MiB;
constexpr unsigned TILE_BYTES = 128 * 1024;
constexpr unsigned WS_CALL = 0;
constexpr unsigned WS_TMO = 4096;
constexpr unsigned WS_TICK = 8192;
constexpr unsigned WS_JTICK = 12288;
constexpr unsigned WS_FLAG = 16384;
constexpr unsigned ID_IN = 0;
constexpr unsigned ID_RP = 384;
constexpr unsigned ID_LP = 576;
constexpr unsigned ID_DP = 896;
constexpr unsigned WS_XR = 1 * MiB;
constexpr unsigned XARR = 8 * MiB, XR_PAR = 24 * MiB;
constexpr unsigned WS_WR = WS_XR + 48 * MiB;
constexpr unsigned WS_LP = WS_WR + 48 * MiB;
constexpr unsigned WS_DP = WS_LP + 40 * MiB;
constexpr unsigned WS_RP = WS_DP + 32 * MiB;
constexpr unsigned RP_PAR = 24 * MiB;
constexpr unsigned WS_END = WS_RP + 48 * MiB;
constexpr unsigned WS_DBG = WS_END;
constexpr unsigned WS_NEED = WS_END + 1 * MiB;

constexpr int HTB = 16384;
constexpr int LDS_STAGE = 8 * HTB, LDS_BYTES = LDS_STAGE + 16;
constexpr unsigned long long DEADLINE_TICKS = 50000000ull;
constexpr int MAXITEMS = 96, MAXJOBS = 64, NCOMPUTE = 28;

struct Link {
    char* ws;
    int nitems;
    int pace, pace2, knee, slack, pad;
};
struct Params {
    const char* x; const char* w; char* ws; int rank, njobs;
    char* out; char* pad[3];
    Link link[NRANK];
    unsigned char zorder[64];
    unsigned short item[8][MAXITEMS];
    unsigned job[8][MAXJOBS];
};
static_assert(sizeof(Params) <= 4096, "kernel arguments");

typedef float f32x2 __attribute__((ext_vector_type(2)));
typedef __bf16 bf16x2n __attribute__((ext_vector_type(2)));
__device__ __forceinline__ unsigned cvt_pk_bf16(float lo, float hi) {
    return __builtin_bit_cast(unsigned, __builtin_convertvector((f32x2){lo, hi}, bf16x2n));
}
__device__ __forceinline__ float bf_lo(unsigned u) { return __uint_as_float(u << 16); }
__device__ __forceinline__ float bf_hi(unsigned u) { return __uint_as_float(u & 0xffff0000u); }
__device__ __forceinline__ f32x4 up_lo(u32x4 c) { return (f32x4){bf_lo(c.x), bf_hi(c.x), bf_lo(c.y), bf_hi(c.y)}; }
__device__ __forceinline__ f32x4 up_hi(u32x4 c) { return (f32x4){bf_lo(c.z), bf_hi(c.z), bf_lo(c.w), bf_hi(c.w)}; }
__device__ __forceinline__ u32x4 pack8(f32x4 a, f32x4 b) {
    return (u32x4){cvt_pk_bf16(a[0], a[1]), cvt_pk_bf16(a[2], a[3]), cvt_pk_bf16(b[0], b[1]), cvt_pk_bf16(b[2], b[3])};
}
__device__ __forceinline__ float silu(float y) {
    const float c = __builtin_amdgcn_fmed3f(y, -60.0f, 60.0f);
    return y * __builtin_amdgcn_rcpf(1.0f + __builtin_amdgcn_exp2f(c * -1.44269504088896341f));
}
__device__ __forceinline__ u32x4 silu_pack8(f32x4 a, f32x4 b, bool bad) {
    const unsigned ax = cvt_pk_bf16(silu(a[0]), silu(a[1])), ay = cvt_pk_bf16(silu(a[2]), silu(a[3]));
    const unsigned bx = cvt_pk_bf16(silu(b[0]), silu(b[1])), by = cvt_pk_bf16(silu(b[2]), silu(b[3]));
    const auto sx = __builtin_amdgcn_permlane16_swap(ax, bx, false, false);
    const auto sy = __builtin_amdgcn_permlane16_swap(ay, by, false, false);
    u32x4 o = {sx[0], sy[0], sx[1], sy[1]};
    if (bad) o = (u32x4){0x7fc07fc0u, 0x7fc07fc0u, 0x7fc07fc0u, 0x7fc07fc0u};
    return o;
}
__device__ __forceinline__ unsigned ld_sys(const char* p) {
    return __hip_atomic_load((const gu32*)p, __ATOMIC_RELAXED, __HIP_MEMORY_SCOPE_SYSTEM);
}
__device__ __forceinline__ unsigned ld_agent(const char* p) {
    return __hip_atomic_load((const gu32*)p, __ATOMIC_RELAXED, __HIP_MEMORY_SCOPE_AGENT);
}
__device__ __forceinline__ unsigned grow_of(int g, int lr) {
    if (lr < 4) return (unsigned)((g ^ 1) * 1024 + lr * 256);
    if (lr >= 12) return (unsigned)(g * 1024 + (lr - 12) * 256);
    const int r = lr - 4;
    return (unsigned)((g ^ (4 + (r & 3))) * 1024 + ((g >> 1) & 1) * 512 + (r >> 2) * 256);
}
__device__ __forceinline__ unsigned khalf(int g, int j) {
    return (unsigned)((g ^ (((g ^ j) >> 2) & (g >> 1) & j)) & 1);
}
__device__ __forceinline__ bool give_up(char* ws, unsigned call, unsigned long long t0);
__device__ __forceinline__ bool wave_wait(char* ws, const char* fp, unsigned call) {
    const unsigned long long t0 = __builtin_amdgcn_s_memrealtime();
    for (unsigned polls = 1;; ++polls) {
        const unsigned f = ld_sys(fp), tm = ld_agent(ws + WS_TMO);
        if (__builtin_amdgcn_readfirstlane(tm) == call) return true;
        if (__all(f >= call)) return false;
        if ((polls & 15u) == 0u && give_up(ws, call, t0)) return true;
        __builtin_amdgcn_s_sleep(4);
    }
}
__device__ __forceinline__ bool give_up(char* ws, unsigned call, unsigned long long t0) {
    const unsigned t = __builtin_amdgcn_readfirstlane(ld_agent(ws + WS_TMO));
    if (t == call) return true;
    if (__builtin_amdgcn_s_memrealtime() - t0 > DEADLINE_TICKS) {
        __hip_atomic_store((gu32*)(ws + WS_TMO), call, __ATOMIC_RELAXED, __HIP_MEMORY_SCOPE_AGENT);
        return true;
    }
    return false;
}

#define STAMP(i) do {} while (0)
#define JSTAMP(i) do {} while (0)

#define WAIT_V(n) asm volatile("s_waitcnt vmcnt(" #n ")" ::: "memory")
#define WAIT_L(n) asm volatile("s_waitcnt lgkmcnt(" #n ")" ::: "memory")
#define BAR asm volatile("s_barrier" ::: "memory")
#define SCHED __builtin_amdgcn_sched_barrier(0)
#define RFL(x) __builtin_amdgcn_readfirstlane(x)

#define CMAP 1
#define LOADF "sc0 sc1"
#define FARSLEEP 48
#define CWIN 36
constexpr unsigned OOB = 0x40000000u;
__device__ __forceinline__ void comm_role(const Params& p, const int cid) {
    const int d = cid < 24 ? 2 * (cid >> 3) + 2 + (cid & 1) : ((cid & 2) ? 8 : 1), cix = cid < 24 ? (cid & 7) >> 1 : (cid & 1) + ((cid >> 1) & 2);
    if ((unsigned)d >= (unsigned)NRANK || threadIdx.x >= 64) return;
    const int lane = threadIdx.x;
    unsigned iw0, iw1;
    asm volatile("global_load_ushort %0, %2, %4\n\tglobal_load_ushort %1, %3, %4" : "=&v"(iw0), "=&v"(iw1)
                 : "v"(lane * 2), "v"(lane < MAXITEMS - 64 ? 128 + lane * 2 : 0), "s"(&((const __attribute__((address_space(4))) Params*)__builtin_amdgcn_kernarg_segment_ptr())->item[d][0]) : "memory");
    const Link lk = p.link[d];
    const char* px = p.x; const char* pw = p.w;
    char* ws = p.ws;
    const int g = p.rank & (NRANK - 1), peer = g ^ d;
    const unsigned kh = khalf(peer, g), ps1 = (peer >> 2) & 1, slot = (unsigned)(d - 2);
    const unsigned lane_lin = (unsigned)lane * 16u, lane_w = (unsigned)(lane >> 5) * WROW + (unsigned)(lane & 31) * 16u;
    const unsigned long long t0 = __builtin_amdgcn_s_memrealtime();
    asm volatile("" :: "s"(lk.ws), "s"(lk.nitems), "s"(lk.pace), "s"(lk.pace2), "s"(lk.knee), "s"(lk.slack), "s"(pw));
    u32x4 X[4][8];
    {
        const u32x4 rx = {(unsigned)(size_t)px, (unsigned)((size_t)px >> 32), IN_BYTES, 0x00020000u};
        unsigned so = d >= 2 ? (grow_of(peer, 0) + (unsigned)cix * 32u) * XROW + kh * 1024u : OOB;
#define LD1(k) "buffer_load_dwordx4 %" #k ", %9, %10, %8 offen\n\ts_add_u32 %8, %8, %11\n\t"
#pragma unroll
        for (int s = 0; s < 4; ++s)
            asm volatile(LD1(0) LD1(1) LD1(2) LD1(3) LD1(4) LD1(5) LD1(6) LD1(7)
                : "=&v"(X[s][0]), "=&v"(X[s][1]), "=&v"(X[s][2]), "=&v"(X[s][3]), "=&v"(X[s][4]), "=&v"(X[s][5]), "=&v"(X[s][6]), "=&v"(X[s][7]), "+s"(so)
                : "v"(lane_lin), "s"(rx), "i"(XROW) : "memory", "scc");
    }
    const unsigned call = RFL(ld_agent(ws + WS_CALL + cid * 4)) + 1u;
    asm volatile("s_waitcnt vmcnt(0)" : "+v"(iw0), "+v"(iw1) :: "memory");
    const unsigned par = call & 1u;
    const int nit = lk.nitems, nun = nit * 4;
    gu32* tick = (gu32*)(ws + WS_TICK + d * 256 + par * 128u);
    if (lane == 0) {
        __hip_atomic_store((gu32*)(ws + WS_CALL + cid * 4), call, __ATOMIC_RELAXED, __HIP_MEMORY_SCOPE_AGENT);
        if (cix == 0)
            __hip_atomic_store((gu32*)(ws + WS_TICK + d * 256 + (par ^ 1u) * 128u), 0u, __ATOMIC_RELAXED, __HIP_MEMORY_SCOPE_AGENT);
    }
    unsigned dsrc[2], ddst[2], dfg[2];
#pragma unroll
    for (int h = 0; h < 2; ++h) {
        const unsigned iw = h ? iw1 : iw0, kind = iw >> 8;
        const unsigned it = (iw & 255u) | (kind == 1u ? (unsigned)((peer >> 1) & 1) : 0u);
        const unsigned blk = (it >> 1) & 15u, hh = it & 1u;
        dfg[h] = ID_IN + slot * 64u + it;
        if (kind == 2u) {
            const unsigned lr = it >> 4, lc = it & 15u;
            const unsigned dt = lr < 4u ? it : 64u + (unsigned)(g & 1) * 64u + ((unsigned)((g >> 1) & 1) * 2u + ((lr - 4u) >> 2)) * 16u + lc;
            dsrc[h] = WS_LP + it * TILE_BYTES;
            ddst[h] = (WS_RP + par * RP_PAR + dt * TILE_BYTES) | 2u;
            dfg[h] = (ID_RP + dt) | (ID_LP + it) << 16;
        } else if (kind == 3u) {
            const int j = ((g & 4) ^ 4) | (int)((it >> 4) & 1u) << 1 | (d & 1);
            const unsigned c = it & 15u, h2 = (unsigned)((g >> 1) & 1), sg = (unsigned)((j ^ g) - 2), sp = (unsigned)((j ^ peer) - 2);
            dsrc[h] = WS_WR + (sg * 512u + h2 * 256u) * WROW + par * 8192u + c * 512u;
            ddst[h] = (WS_WR + (sp * 512u + h2 * 256u) * WROW + par * 8192u + c * 512u) | 3u;
            dfg[h] = (ID_IN + sp * 64u + (16u + c) * 2u + h2) | (ID_IN + sg * 64u + (16u + c) * 2u + h2) << 16;
        } else if (it & 32u) {
            dsrc[h] = (kh * 512u + hh * 256u) * WROW + ps1 * 8192u + blk * 512u;
            ddst[h] = (WS_WR + (slot * 512u + hh * 256u) * WROW + par * 8192u + blk * 512u) | 1u;
        } else {
            dsrc[h] = (grow_of(peer, (int)blk) + hh * 128u) * XROW + kh * 1024u;
            ddst[h] = WS_XR + par * XR_PAR + (slot >> 1) * XARR + (blk * 256u + hh * 128u) * XROW + (slot & 1u) * 1024u;
        }
    }
    char* pb = lk.ws;
    const u32x4 rd = {(unsigned)(size_t)pb, (unsigned)((size_t)pb >> 32), WS_END, 0x00020000u};
    gu32* flags = (gu32*)(pb + WS_FLAG);
    const u32x4 rl = {(unsigned)(size_t)ws, (unsigned)((size_t)ws >> 32), WS_XR, 0x00020000u};
    const int pace = lk.pace, pace2 = lk.pace2, knee = lk.knee, slack = lk.slack;
    const int ufirst = d >= 2 ? 4 : 0;
    int ub = ufirst + cix;
    unsigned long long rmask = 0;
    u32x4 R = {0u, 0u, 0u, 0u};
    unsigned T = 0, voa = lane_lin, dofa = OOB, stra = XROW;
    int fa = -1, fprev = -1, duea = 0, pa = pace, tbase = 0;
    if (d >= 2) { dofa = (unsigned)__builtin_amdgcn_readlane((int)ddst[0], 0) + (unsigned)cix * 32u * XROW; fa = (int)(ID_IN + slot * 64u) * 4 + cix; }
    bool seen = false;
    const unsigned one = 1u;
    constexpr int LAG = (CWIN + 14) / 16;
    static_assert(CWIN <= 48 && LAG <= 3, "window");
#define MV1(k) "s_waitcnt vmcnt(%16)\n\tbuffer_store_dwordx4 %" #k ", %10, %11, %8 offen sc0 sc1\n\ts_add_u32 %8, %8, %12\n\t" \
        "buffer_load_dwordx4 %" #k ", %13, %14, %9 offen " LOADF "\n\ts_add_u32 %9, %9, %15\n\t"
#define PUBLISH(fw) do { if (lane == 0) __hip_atomic_store(flags + ((fw) & 0xffff), call, __ATOMIC_RELAXED, __HIP_MEMORY_SCOPE_SYSTEM); CSTAMP(fw); } while (0)
#define CSTAMP(fw) do {} while (0)
#define GATES(go, itn) const int _c = ((lane - (itn)) & 63) + (itn); const unsigned _gid = (_c < 64 ? dfg[0] : dfg[1]) >> 16; \
        const bool look = _c < (itn) + 8 && _c < nit && _gid != 0u; const unsigned go = look ? WS_FLAG + _gid * 16u : OOB
#define OPEN(r) __ballot((r).x >= call && (r).y >= call && (r).z >= call && (r).w >= call)
    for (;;) {
        const int itn = ub >> 2;
        const bool live = ub < nun;
        if (!live && fa < 0) break;
        const unsigned fg = (unsigned)(itn < 64 ? __builtin_amdgcn_readlane((int)dfg[0], itn & 63) : __builtin_amdgcn_readlane((int)dfg[1], itn & 63));
        if (seen) rmask = OPEN(R);
        bool real = live && ((fg >> 16) == 0u || ((rmask >> (itn & 63)) & 1ull));
        GATES(go, itn);
        if (live && !real && fa < 0) {
            WAIT_V(0);
            if (fprev >= 0) { PUBLISH(fprev); fprev = -1; }
            for (unsigned polls = 1;; ++polls) {
                asm volatile("buffer_load_dwordx4 %0, %1, %2, 0 offen sc0 sc1\n\ts_waitcnt vmcnt(0)" : "+v"(R) : "v"(go), "s"(rl) : "memory");
                rmask = OPEN(R);
                if ((rmask >> (itn & 63)) & 1ull) break;
                if ((polls & 63u) == 0u && give_up(ws, call, t0)) return;
                __builtin_amdgcn_s_sleep(2);
            }
            real = true;
        }
        const unsigned so = (unsigned)(itn < 64 ? __builtin_amdgcn_readlane((int)dsrc[0], itn & 63) : __builtin_amdgcn_readlane((int)dsrc[1], itn & 63));
        const unsigned dk = (unsigned)(itn < 64 ? __builtin_amdgcn_readlane((int)ddst[0], itn & 63) : __builtin_amdgcn_readlane((int)ddst[1], itn & 63));
        const unsigned kind = dk & 3u;
        const unsigned strb = RFL(kind == 0u ? XROW : ((kind & 1u) ? 2u * WROW : 1024u));
        const char* sb = kind == 0u ? px : (kind == 1u ? pw : ws);
        const u32x4 rs = {(unsigned)(size_t)sb, (unsigned)((size_t)sb >> 32), sb == ws ? WS_END : IN_BYTES, 0x00020000u};
        const unsigned uofs = (unsigned)((ub & 3) * 32) * strb;
        const unsigned vob = (kind & 1u) ? lane_w : lane_lin;
        unsigned sof = RFL(real ? so + uofs : OOB), dof = RFL(dofa);
        const unsigned sta = RFL(stra);
        if (real && lane == 0) asm volatile("global_atomic_add %0, %1, %2, off sc0" : "+v"(T) : "v"(tick), "v"(one) : "memory");
        seen = false;
#pragma unroll
        for (int s = 0; s < 4; ++s) {
            if (fa >= 0) {
                int lead;
                while ((lead = tbase + duea + s * pa - (int)(unsigned)(__builtin_amdgcn_s_memrealtime() - t0) * 16) > 0) { }
                if (lead < -slack) tbase -= lead + slack;
            }
            if (s == LAG && fprev >= 0) { asm volatile("s_waitcnt vmcnt(%0)" :: "i"(CWIN) : "memory"); PUBLISH(fprev); }
            asm volatile("s_nop 4\n\t" MV1(0) MV1(1) MV1(2) MV1(3) MV1(4) MV1(5) MV1(6) MV1(7)
                : "+v"(X[s][0]), "+v"(X[s][1]), "+v"(X[s][2]), "+v"(X[s][3]), "+v"(X[s][4]), "+v"(X[s][5]), "+v"(X[s][6]), "+v"(X[s][7]), "+s"(dof), "+s"(sof)
                : "v"(voa), "s"(rd), "s"(sta), "v"(vob), "s"(rs), "s"(strb), "i"(CWIN) : "memory", "scc");
            if (s == 0 && live && __any(look)) {
                asm volatile("buffer_load_dwordx4 %0, %1, %2, 0 offen sc0 sc1" : "+v"(R) : "v"(go), "s"(rl) : "memory");
                seen = true;
            }
        }
        asm volatile("s_waitcnt vmcnt(48)" : "+v"(R), "+v"(T) :: "memory");
        fprev = fa;
        fa = real ? (int)((fg & 0xffffu) * 4u + (unsigned)(ub & 3)) | itn << 16 : -1;
        voa = vob; stra = strb; dofa = real ? (dk & ~3u) + uofs : OOB;
        pa = itn < knee ? pace : pace2;
        duea = (itn < knee ? itn * pace : knee * pace + (itn - knee) * pace2) * 4 + (((ub & 3) * pa) >> 2);
        if (real) ub = RFL((int)T) + ufirst + 4;
    }
    WAIT_V(0);
    if (fprev >= 0) PUBLISH(fprev);
}

__global__ __launch_bounds__(512, 2) void gemm_rs_kernel(const Params p) {
    extern __shared__ __attribute__((aligned(16))) unsigned char shm[];
    LAS unsigned char* lds = (LAS unsigned char*)shm;
    const int g = p.rank;
    char* ws = p.ws;
    const int bid = blockIdx.x;
    STAMP(0);
    if (__builtin_expect(bid < 32, 1)) {
        comm_role(p, bid);
    } else {
        const unsigned call = RFL(ld_agent(ws + WS_CALL + bid * 4)) + 1u;
        const int tid = threadIdx.x, wid = __builtin_amdgcn_readfirstlane(tid >> 6), lane = tid & 63;
        const int wr = wid >> 2, wc = wid & 3, fr = lane & 15, fq = lane >> 4;
        const unsigned par = call & 1u;
        const int s1 = (g >> 2) & 1;
        unsigned voffA, voffB;
        { const int r = wid * 8 + (lane >> 3); voffA = (unsigned)r * XROW + (unsigned)(((lane & 7) ^ ((r >> 1) & 7)) * 16); }
        { const int r = wid * 4 + (lane >> 4); voffB = (unsigned)r * WROW + (unsigned)(((lane & 15) ^ (((r & 3) << 2) | (((r >> 3) & 1) << 1))) * 16); }
        const unsigned ldsw = (unsigned)(size_t)lds + (unsigned)wid * 1024u;
        const int aoff0 = (wr * 64 + fr) * 128 + ((fq ^ (fr >> 1)) * 16), aoff1 = aoff0 ^ 64;
        int boff0, boff1;
        { const int q = (lane & 15) >> 2, pp = lane & 3, r = 8 * fq + q;
          boff0 = 4 * HTB + r * 256 + (((wc * 4 + (pp >> 1)) ^ ((q << 2) | ((fq & 1) << 1))) * 16) + (pp & 1) * 8; boff1 = boff0 ^ 32; }
        asm volatile("" : "+v"(boff0), "+v"(boff1));
#define SA(b, h) (((b) * 2 + (h)) * HTB)
#define SB(b, h) (((b) * 2 + (h)) * HTB)
#define DMA2(rs, voff, so, l0, o0, o1) do { unsigned _t; asm volatile( \
        "s_add_u32 m0, %3, %5\n\ts_add_u32 %0, %4, %7\n\tbuffer_load_dwordx4 %1, %2, %0 offen lds\n\t" \
        "s_add_u32 m0, %3, %6\n\ts_add_u32 %0, %4, %8\n\tbuffer_load_dwordx4 %1, %2, %0 offen lds" \
        : "=&s"(_t) : "v"(voff), "s"(rs), "s"(ldsw), "s"(so), "i"(l0), "i"((l0) + 8192), "i"(o0), "i"(o1) : "memory", "scc"); } while (0)
#define STAGE_A(b, h, so, odd) DMA2(ra, voffA, so, SA(b, h), (h) * 128 * XROW + (odd) * 128, ((h) * 128 + 64) * XROW + (odd) * 128)
#define STAGE_B(b, h, so, odd) DMA2(rb, voffB, so, 4 * HTB + SB(b, h), (h) * 256 + (odd) * 64 * WROW, (h) * 256 + ((odd) * 64 + 32) * WROW)
#define LDA(dst, b, h) do { _Pragma("unroll") for (int m = 0; m < 4; ++m) { \
        dst[m][0] = *(const LAS bf16x8*)(lds + SA(b, h) + aoff0 + m * 2048); dst[m][1] = *(const LAS bf16x8*)(lds + SA(b, h) + aoff1 + m * 2048); } } while (0)
#define TR(off) __builtin_amdgcn_ds_read_tr16_b64_v4i16((LAS bf16x4*)(lds + (off)))
#define LDB(dst, b, h) do { _Pragma("unroll") for (int k = 0; k < 2; ++k) { \
        dst[0][k] = __builtin_shufflevector(TR(SB(b, h) + boff0 + k * 8192), TR(SB(b, h) + boff0 + k * 8192 + 1024), 0, 1, 2, 3, 4, 5, 6, 7); \
        dst[1][k] = __builtin_shufflevector(TR(SB(b, h) + boff1 + k * 8192), TR(SB(b, h) + boff1 + k * 8192 + 1024), 0, 1, 2, 3, 4, 5, 6, 7); } } while (0)
#define NEXTJOB() ({ if (threadIdx.x == 0 && !have) { lctl[0] = __hip_atomic_fetch_add((gu32*)(ws + WS_JTICK + xq * 256 + par * 128u), 1u, __ATOMIC_RELAXED, __HIP_MEMORY_SCOPE_AGENT) + NCOMPUTE; WAIT_L(0); } \
        have = false; BAR; jx = RFL(lctl[0]); WAIT_L(0); BAR; })
#define WG_WAIT(fp) ({ if (wid == 0) { if (wave_wait(ws, fp, call) && threadIdx.x == 0) lctl[1] = 1u; WAIT_L(0); } BAR; RFL(lctl[1]) != 0u; })
#define PIN(ai, bj) asm volatile("" : "+v"(acc[ai][bj][0][0]), "+v"(acc[ai][bj][0][1]), "+v"(acc[ai][bj][1][0]), "+v"(acc[ai][bj][1][1]), \
        "+v"(acc[ai][bj][2][0]), "+v"(acc[ai][bj][2][1]), "+v"(acc[ai][bj][3][0]), "+v"(acc[ai][bj][3][1]))
#define MMA(ai, bj, At, Bt) do { __builtin_amdgcn_s_setprio(1); PIN(ai, bj); \
        _Pragma("unroll") for (int m = 0; m < 4; ++m) _Pragma("unroll") for (int n = 0; n < 2; ++n) _Pragma("unroll") for (int k = 0; k < 2; ++k) \
        acc[ai][bj][m][n] = __builtin_amdgcn_mfma_f32_16x16x32_bf16(Bt[n][k], At[m][k], acc[ai][bj][m][n], 0, 0, 0); \
        PIN(ai, bj); __builtin_amdgcn_s_setprio(0); } while (0)

        const int xq = bid & 7;
        LAS unsigned* lctl = (LAS unsigned*)(lds + LDS_STAGE);
        if (tid == 0) {
            lctl[1] = 0u; lctl[3] = 0u;
            if (bid < 40) __hip_atomic_store((gu32*)(ws + WS_JTICK + xq * 256 + (par ^ 1u) * 128u), 0u, __ATOMIC_RELAXED, __HIP_MEMORY_SCOPE_AGENT);
        }
        const int njobs = p.njobs;
        int jx = (bid - 32) >> 3;
        bool have = false;
        for (; jx < njobs; NEXTJOB()) {
            const unsigned job = p.job[xq][jx];
            const int type = (job >> 8) & 7, lr = (job >> 4) & 15, lc = job & 15;
            JSTAMP(0);
            f32x4 acc[2][2][4][2];
            {
                const unsigned dt = 64u + (unsigned)((lr - 12) * 16 + lc);
                unsigned ib = WS_END;
                if (type == 3 && !(job >> 31)) { (void)WG_WAIT(ws + WS_FLAG + (ID_DP + dt) * 16u); ib = WS_DP + dt * (2u * TILE_BYTES) + (unsigned)(wid * 32768 + lane * 16); }
                const auto rl = __builtin_amdgcn_make_buffer_rsrc((void*)ws, 0, WS_END, 0x00020000);
#pragma unroll
                for (int i = 0; i < 32; ++i)
                    acc[i >> 4][(i >> 3) & 1][(i >> 1) & 3][i & 1] = __builtin_bit_cast(f32x4, __builtin_amdgcn_raw_buffer_load_b128(rl, ib + (unsigned)(i * 1024), 0, 16));
                PIN(0, 0); PIN(0, 1); PIN(1, 0); PIN(1, 1);
            }
            for (int ph = (job >> 31) ? 0 : 1; ph < 3; ++ph) {
                const char* abase; const char* bbase; unsigned a0, b0, nbytes, segs; int nt;
                if (ph == 0) {
                    const bool other = type == 4 && lr < 4;
                    const unsigned grow = grow_of(g, other ? lr + 12 : lr);
                    const unsigned gcol = (unsigned)((other ? 1 - s1 : s1) * 4096 + lc * 256);
                    abase = p.x; bbase = p.w; nbytes = IN_BYTES;
                    a0 = grow * XROW; b0 = gcol * 2u; nt = 16; segs = 0x10u;
                } else {
                    const unsigned mask = (job >> (5 + 6 * ph)) & 63u;
                    if (mask == 0u) break;
                    abase = ws; bbase = ws; nbytes = WS_END;
                    a0 = WS_XR + par * XR_PAR + (unsigned)lr * 256u * XROW; b0 = WS_WR + par * 8192u + (unsigned)lc * 512u;
                    segs = 0u; nt = 0;
                    for (int sl = 0; sl < 6; ++sl) if ((mask >> sl) & 1u) { segs |= (unsigned)sl << (nt >> 1); nt += 8; }
                    JSTAMP(2 * ph - 1);
                    if (wid == 0) {
                        const unsigned sl = ((mask >> (lane >> 3)) & 1u) ? (unsigned)(lane >> 3) : (segs & 15u);
                        const char* fx = ws + WS_FLAG + (ID_IN + sl * 64u + (unsigned)lr * 2u) * 16u + (lane & 7) * 4;
                        const char* fw = ws + WS_FLAG + (ID_IN + sl * 64u + (unsigned)(16 + lc) * 2u) * 16u + (lane & 7) * 4;
                        const unsigned long long t0 = __builtin_amdgcn_s_memrealtime();
                        {
                            const char* fe = ws + WS_FLAG + (ID_IN + (segs & 15u) * 64u + (unsigned)(lane < 32 ? lr : 16 + lc) * 2u) * 16u + (lane & 7) * 4;
                            for (unsigned polls = 1;; ++polls) {
                                const unsigned long long on = __ballot(ld_sys(fe) >= call);
                                if ((unsigned)on != 0u && (unsigned)(on >> 32) != 0u) break;
                                if ((polls & 15u) == 0u && give_up(ws, call, t0)) break;
                                __builtin_amdgcn_s_sleep(FARSLEEP);
                            }
                        }
                        for (unsigned polls = 1;; ++polls) {
                            const unsigned f0 = ld_sys(fx), f1 = ld_sys(fw);
                            if (__all(f0 >= call && f1 >= call)) break;
                            if ((polls & 15u) == 0u && give_up(ws, call, t0)) { if (lane == 0) lctl[1] = 1u; break; }
                            __builtin_amdgcn_s_sleep(6);
                        }
                        __builtin_amdgcn_fence(__ATOMIC_ACQUIRE, "");
                        WAIT_V(0);
                    }
                    BAR;
                    JSTAMP(2 * ph);
                }
                bf16x8 At[4][2], B0[2][2], B1[2][2];
                const u32x4 ra = {(unsigned)(size_t)abase, (unsigned)((size_t)abase >> 32), nbytes, 0x00020000u};
                const u32x4 rb = {(unsigned)(size_t)bbase, (unsigned)((size_t)bbase >> 32), nbytes, 0x00020000u};
#define KOFF(t, oa, ob) do { const int _s = (segs >> (((t) >> 3) * 4)) & 15u; \
                oa = a0 + (unsigned)(_s >> 1) * XARR + (unsigned)(_s & 1) * 1024u + (unsigned)((t) & 7) * 128u; \
                ob = b0 + (unsigned)(_s * 8 + ((t) & 7)) * (64u * WROW); } while (0)
                unsigned ca, cb;
                KOFF(0, ca, cb);
                STAGE_B(0, 0, cb, 0); STAGE_A(0, 0, ca, 0); STAGE_B(0, 1, cb, 0); STAGE_A(0, 1, ca, 0);
                if (wr == 1) BAR;
                WAIT_V(4); BAR;
                STAGE_B(1, 0, cb, 1); STAGE_A(1, 0, ca, 1); STAGE_B(1, 1, cb, 1);
                WAIT_V(6); BAR;
                for (int t = 0; t < nt; t += 2) {
                    const int tn = t + 2 < nt ? t + 2 : 0;
                    unsigned a2, b2;
                    KOFF(tn, a2, b2);
                    LDB(B0, 0, 0); SCHED; LDA(At, 0, 0); STAGE_A(1, 1, ca, 1);
                    WAIT_L(8); BAR; WAIT_L(0); MMA(0, 0, At, B0); BAR; SCHED;
                    LDB(B1, 0, 1); STAGE_B(0, 0, b2, 0);
                    BAR; WAIT_L(0); MMA(0, 1, At, B1); BAR;
                    LDA(At, 0, 1); STAGE_A(0, 0, a2, 0);
                    BAR; WAIT_L(0); MMA(1, 0, At, B0); BAR; SCHED;
                    STAGE_B(0, 1, b2, 0);
                    WAIT_V(6); BAR; MMA(1, 1, At, B1); BAR;
                    LDB(B0, 1, 0); SCHED; LDA(At, 1, 0); STAGE_A(0, 1, a2, 0);
                    WAIT_L(8); BAR; WAIT_L(0); MMA(0, 0, At, B0); BAR; SCHED;
                    LDB(B1, 1, 1); STAGE_B(1, 0, b2, 1);
                    BAR; WAIT_L(0); MMA(0, 1, At, B1); BAR;
                    LDA(At, 1, 1); STAGE_A(1, 0, a2, 1);
                    BAR; WAIT_L(0); MMA(1, 0, At, B0); BAR; SCHED;
                    STAGE_B(1, 1, b2, 1);
                    WAIT_V(6); BAR; MMA(1, 1, At, B1); BAR;
                    ca = a2;
                }
                WAIT_V(0);
                if (wr == 0) BAR;
                BAR;
            }
            JSTAMP(5);
            int tz = threadIdx.x; asm volatile("" : "+v"(tz));
            const int elane = tz & 63;
            const unsigned toff = (unsigned)wid * 16384u + (unsigned)elane * 16u;
            const auto rl = __builtin_amdgcn_make_buffer_rsrc((void*)ws, 0, WS_END, 0x00020000);
            const unsigned ht = 192u + ((job >> 23) & 127u);
            const unsigned tl = type == 2 ? ht : (unsigned)((lr < 12 ? lr : lr - 8) * 16 + lc);
            const unsigned pbase = WS_LP + tl * TILE_BYTES + toff;
            bool bad = false;
            if (type == 3 || ((job >> 30) & 1u)) {
                const unsigned t1 = (unsigned)((lr - 12) * 16 + lc);
                bad = WG_WAIT(ws + WS_FLAG + (type == 3 ? ID_RP + t1 : ID_LP + ht) * 16u + (elane & 3) * 4);
                JSTAMP(6);
                const unsigned base1 = bad ? WS_END : (type == 3 ? WS_RP + par * RP_PAR + t1 * TILE_BYTES : WS_LP + ht * TILE_BYTES) + toff;
                u32x4 q[16];
#pragma unroll
                for (int i = 0; i < 16; ++i) q[i] = __builtin_amdgcn_raw_buffer_load_b128(rl, base1 + (unsigned)(i * 1024), 0, 17);
                if (type == 3) {
                    if (tz == 0) { lctl[0] = __hip_atomic_fetch_add((gu32*)(ws + WS_JTICK + xq * 256 + par * 128u), 1u, __ATOMIC_RELAXED, __HIP_MEMORY_SCOPE_AGENT) + NCOMPUTE; WAIT_L(0); }
                    have = true;
                }
                SCHED;
#pragma unroll
                for (int i = 0; i < 16; ++i) {
                    const int a = i >> 3, b = (i >> 2) & 1, m = i & 3;
                    acc[a][b][m][0] += up_lo(q[i]); acc[a][b][m][1] += up_hi(q[i]);
                    asm volatile("" : "+v"(acc[a][b][m][0]), "+v"(acc[a][b][m][1]));
                }
            }
            if (type == 3) {
                const auto ro = __builtin_amdgcn_make_buffer_rsrc((void*)p.out, 0, OUT_BYTES, 0x00020000);
                const unsigned obase = (unsigned)((lr - 12) * 256 + wr * 64 + (elane & 15)) * WROW
                                     + (unsigned)(s1 * 4096 + lc * 256 + wc * 32 + ((elane >> 4) & 1) * 16 + (elane >> 5) * 8) * 2u;
#pragma unroll
                for (int i = 0; i < 16; ++i) {
                    const int a = i >> 3, b = (i >> 2) & 1, m = i & 3;
                    __builtin_amdgcn_raw_buffer_store_b128(silu_pack8(acc[a][b][m][0], acc[a][b][m][1], bad), ro,
                                                           obase + (unsigned)(a * 128 + m * 16) * WROW + (unsigned)(b * 256), 0, 0);
                }
            } else if (type == 4) {
#pragma unroll
                for (int i = 0; i < 16; ++i)
#pragma unroll
                    for (int n = 0; n < 2; ++n)
                        __builtin_amdgcn_raw_buffer_store_b128(__builtin_bit_cast(u32x4, acc[i >> 3][(i >> 2) & 1][i & 3][n]), rl,
                            WS_DP + tl * (2u * TILE_BYTES) + (unsigned)wid * 32768u + (unsigned)elane * 16u + (unsigned)((i * 2 + n) * 1024), 0, 16);
            } else {
#pragma unroll
                for (int i = 0; i < 16; ++i)
                    __builtin_amdgcn_raw_buffer_store_b128(pack8(acc[i >> 3][(i >> 2) & 1][i & 3][0], acc[i >> 3][(i >> 2) & 1][i & 3][1]), rl, pbase + (unsigned)(i * 1024), 0, 16);
            }
            if (type != 3) {
                asm volatile("s_waitcnt vmcnt(0) lgkmcnt(0)" ::: "memory");
                BAR;
                if (tz < 4 && lctl[1] == 0u)
                    __hip_atomic_store((gu32*)(ws + WS_FLAG + ((type == 4 ? ID_DP : ID_LP) + tl) * 16u + tz * 4), call, __ATOMIC_RELAXED, __HIP_MEMORY_SCOPE_AGENT);
            }
            JSTAMP(7);
        }
        for (; jx < njobs + 32; NEXTJOB()) {
            JSTAMP(0);
            const unsigned z = (unsigned)((jx - njobs) * 8 + xq), t = p.zorder[z >> 2], u = z & 3u;
            const int ow = (int)(2u * u) + (wid >> 2), grp = wid & 3;
            const auto rl = __builtin_amdgcn_make_buffer_rsrc((void*)ws, 0, WS_END, 0x00020000);
            (void)WG_WAIT(ws + WS_FLAG + (ID_DP + t) * 16u);
            const char* jt = ws + WS_JTICK + xq * 256 + par * 128u;
            u32x4 dq[8], q1[4], q2[4];
#pragma unroll
            for (int i = 0; i < 8; ++i)
                dq[i] = __builtin_amdgcn_raw_buffer_load_b128(rl, WS_DP + t * (2u * TILE_BYTES) + (unsigned)(ow * 32768 + (grp * 8 + i) * 1024 + lane * 16), 0, 17);
            if (wid == 1) {
                const unsigned long long t0 = __builtin_amdgcn_s_memrealtime();
                bool empty;
                do { empty = (int)RFL(ld_agent(jt)) + NCOMPUTE >= njobs + 32; if (!empty) __builtin_amdgcn_s_sleep(32); }
                while (!empty && RFL(*(volatile LAS unsigned*)(lctl + 3)) != z + 1u && __builtin_amdgcn_s_memrealtime() - t0 < 2u * DEADLINE_TICKS);
                if (lane == 0) lctl[2] = empty ? 1u : 0u;
                WAIT_L(0);
            }
            const bool bad = ({ if (wid == 0) { if (wave_wait(ws, ws + WS_FLAG + (ID_RP + 64u + t + (unsigned)(lane >> 5) * 64u) * 16u + u * 4u, call) && lane == 0) lctl[1] = 1u;
                                                if (lane == 0) lctl[3] = z + 1u; WAIT_L(0); } BAR; RFL(lctl[1]) != 0u; });
            JSTAMP(6);
            const unsigned base = bad ? WS_END : WS_RP + par * RP_PAR + (64u + t) * TILE_BYTES + (unsigned)(ow * 16384 + grp * 4096 + lane * 16);
#pragma unroll
            for (int m = 0; m < 4; ++m) {
                q1[m] = __builtin_amdgcn_raw_buffer_load_b128(rl, base + (unsigned)(m * 1024), 0, 17);
                q2[m] = __builtin_amdgcn_raw_buffer_load_b128(rl, base + 64u * TILE_BYTES + (unsigned)(m * 1024), 0, 17);
            }
            const auto ro = __builtin_amdgcn_make_buffer_rsrc((void*)p.out, 0, OUT_BYTES, 0x00020000);
            const unsigned obase = ((t >> 4) * 256u + (unsigned)((grp >> 1) * 128 + (ow >> 2) * 64 + fr)) * WROW
                                 + ((unsigned)((1 - s1) * 4096 + (grp & 1) * 128 + (ow & 3) * 32 + (fq & 1) * 16 + (fq >> 1) * 8) + (t & 15u) * 256u) * 2u;
#pragma unroll
            for (int m = 0; m < 4; ++m) {
                const f32x4 v0 = __builtin_bit_cast(f32x4, dq[2 * m]) + up_lo(q1[m]) + up_lo(q2[m]);
                const f32x4 v1 = __builtin_bit_cast(f32x4, dq[2 * m + 1]) + up_hi(q1[m]) + up_hi(q2[m]);
                __builtin_amdgcn_raw_buffer_store_b128(silu_pack8(v0, v1, bad), ro, obase + (unsigned)(m * 16) * WROW, 0, 0);
            }
            JSTAMP(7);
            if (RFL(lctl[2]) != 0u) break;
        }
        if (tid == 0) __hip_atomic_store((gu32*)(ws + WS_CALL + bid * 4), call, __ATOMIC_RELAXED, __HIP_MEMORY_SCOPE_AGENT);
    }
    STAMP(13);
}

#define PACE1 892
#define PACEL 876
#define KNEE1 64
#define KNEE3 40
#define PACE2 892
#define PACE3 892
#define SLACK 4500
#define SPLIT_COLS 16
#define ORDER1 "x0 w0 w1 x1 w2 x2 w3 x3 x4 w4 x5 w5 x6 x7 w6 x8 w7 x9 w8 x10 w9 x11 w10 w11 w12 w13 w14 w15 x12 x13 x14 x15"
#define ORDER2 ORDER1
int knob(const char* name, int dflt) {
    (void)name; return dflt;
}
void build_schedule(Params& p) {
    const char* ord[2] = {ORDER1, ORDER2};
    const int nsplit = knob("SPLIT", SPLIT_COLS), phased = knob("PHASED", 3);
    for (int d = 0; d < 8; ++d) {
        Link& l = p.link[d];
        l.pace = d >= 4 ? knob("PACE1", PACE1) : (d >= 2 ? knob("PACE2", PACE2) : knob("PACE3", PACE3));
        l.pace2 = knob("PACEL", PACEL); l.knee = d >= 4 ? knob("KNEE1", KNEE1) : (d >= 2 ? 99 : knob("KNEE3", KNEE3));
        l.slack = knob("SLACK", SLACK); l.pad = 0;
    }
    int arr[2][32] = {}, nit[8] = {};
    for (int d = 2; d < 8; ++d)
        for (const char* e = ord[d >= 4 ? 0 : 1]; *e;) {
            const char c = *e;
            if (c != 'x' && c != 'w' && c != 'h' && c != 'r') { ++e; continue; }
            int n = 0; for (++e; *e >= '0' && *e <= '9'; ++e) n = n * 10 + (*e - '0');
            const int b = (c == 'x' ? 0 : 16) + (n & 15);
            if (d >= 4 && b >= 4 && b < 12 && ((b - 4) & 3) == d - 4) continue;
            if (nit[d] + 2 > MAXITEMS) continue;
            if (c == 'h') p.item[d][nit[d]++] = (unsigned short)(1 << 8 | b << 1);
            else for (int h = 0; h < 2; ++h) p.item[d][nit[d]++] = (unsigned short)(c == 'r' ? 3 << 8 | h << 4 | (n & 15) : b << 1 | h);
            int& a = arr[d >= 4 || c == 'r' ? 0 : 1][b];
            if (nit[d] > a) a = nit[d];
        }
    auto mx = [](int a, int b) { return a > b ? a : b; };
    auto t1 = [&](int lr, int lc) { return mx(arr[0][lr], arr[0][16 + lc]); };
    auto t2 = [&](int lr, int lc) { return mx(arr[1][lr], arr[1][16 + lc]); };
    auto tin = [&](int lr, int lc) { return mx(t1(lr, lc), t2(lr, lc)); };
    int zn = 0;
    for (int tm = 0; tm <= MAXITEMS; ++tm)
        for (int lr = 0; lr < 12; ++lr)
            for (int lc = 0; lc < 16; ++lc) {
                if (tin(lr, lc) != tm) continue;
                const int d = lr < 4 ? 1 : 4 + ((lr - 4) & 3);
                if (nit[d] < MAXITEMS) p.item[d][nit[d]++] = (unsigned short)(2 << 8 | (lr * 16 + lc));
                if (d == 7) for (int h = 0; h < 2; ++h) p.zorder[zn++] = (unsigned char)((2 * h + ((lr - 4) >> 2)) * 16 + lc);
            }
    for (int d = 0; d < 8; ++d) { p.link[d].nitems = nit[d]; for (int k = nit[d]; k < MAXITEMS; ++k) p.item[d][k] = 0; }
    auto J = [](int type, int lr, int lc, int ma, int mb, int own) { return (unsigned)(lc | lr << 4 | type << 8 | ma << 11 | mb << 17) | (unsigned)own << 31; };
    const int town = 9, oown = knob("OOWN", 0), mix = knob("MIX", 1);
    for (int x = 0; x < 8; ++x) {
        const int q = x >> 1, hx = x & 1;
        unsigned job[MAXJOBS]; int key[MAXJOBS], n = 0, nh = 0;
        for (int i = 0; i < 4; ++i)
            for (int u = 0; u < 8; ++u) {
                const int lr = 2 * u + (hx ^ (u & mix)), lc = 4 * i + ((i & mix) ? 3 - q : q), all = lr >= 4 && lr < 12 ? 63 & ~(4 << ((lr - 4) & 3)) : 63;
                const int own = lr >= 12 && oown ? 0 : 1;
                if (lr < 4 && 4 * i < nsplit) {
                    const unsigned ht = (unsigned)(nh++ * 8 + x) << 23;
                    job[n] = J(2, lr, lc, 56, 0, 0) | ht; key[n++] = t1(lr, lc) - 1;
                    job[n] = J(1, lr, lc, 7, 0, 1) | ht | 1u << 30; key[n++] = tin(lr, lc) - town;
                } else if (phased & (lr < 12 ? 1 : 2)) {
                    job[n] = J(lr < 12 ? 1 : 3, lr, lc, all & 60, 3, own); key[n++] = t1(lr, lc) - (own ? town : 2);
                } else {
                    job[n] = J(lr < 12 ? 1 : 3, lr, lc, all, 0, own); key[n++] = tin(lr, lc) - (own ? town : 2);
                }
                if (u < 2) { job[n] = J(4, lr, lc, 0, 0, 1); key[n++] = knob("FKEY", 12); }
                if (!own) { job[n] = J(4, lr, lc, 0, 0, 1); key[n++] = knob("OKEY", 12); }
            }
        p.njobs = n;
        for (int k = 0; k < n; ++k) {
            int best = -1;
            for (int j = 0; j < n; ++j) if (key[j] < 1000 && (best < 0 || key[j] < key[best])) best = j;
            p.job[x][k] = job[best]; key[best] = 1000;
        }
        for (int k = n; k < MAXJOBS; ++k) p.job[x][k] = 0;
    }
}

}

extern "C" void kernel_launch(int rank, int nranks, ncclComm_t comm,
                              hipStream_t stream,
                              void* const* d_in, const int* in_sizes, int n_in,
                              void* d_out, int out_size,
                              const int* in_dtypes, int out_dtype,
                              void* const* d_out_all,
                              void* const* d_ws_all, size_t ws_bytes) {
    (void)comm; (void)in_dtypes; (void)out_dtype; (void)d_out_all;
    if (nranks != NRANK || (unsigned)rank >= (unsigned)NRANK || n_in < 2 || ws_bytes < (size_t)WS_NEED ||
        in_sizes[0] != 8192 * 1024 || in_sizes[1] != 1024 * 8192 || out_size != 1024 * 8192) {
        fprintf(stderr, "kernel_launch: unexpected problem shape\n");
        return;
    }
    HIP_CHECK(hipFuncSetAttribute((const void*)gemm_rs_kernel, hipFuncAttributeMaxDynamicSharedMemorySize, LDS_BYTES));
    Params p;
    p.x = (const char*)d_in[0]; p.w = (const char*)d_in[1]; p.out = (char*)d_out;
    for (int i = 0; i < NRANK; ++i) p.link[i].ws = (char*)d_ws_all[rank ^ i];
    p.ws = p.link[0].ws; p.pad[0] = p.pad[1] = p.pad[2] = nullptr;
    p.rank = rank;
    build_schedule(p);
    hipLaunchKernelGGL(gemm_rs_kernel, dim3(256), dim3(512), LDS_BYTES, stream, p);
    HIP_CHECK(hipGetLastError());
}
```

```cpp
#include <hip/hip_runtime.h>
#include <rccl/rccl.h>
#include <cstdio>

#define HIP_CHECK(expr) do { hipError_t _e = (expr); if (_e != hipSuccess) { \
    fprintf(stderr, "HIP error %s at %s:%d\n", hipGetErrorString(_e), __FILE__, __LINE__); \
    return; } } while (0)

namespace {

#define LAS __attribute__((address_space(3)))
#define GAS __attribute__((address_space(1)))
typedef short bf16x8 __attribute__((ext_vector_type(8)));
typedef short bf16x4 __attribute__((ext_vector_type(4)));
typedef float f32x4 __attribute__((ext_vector_type(4)));
typedef unsigned u32x4 __attribute__((ext_vector_type(4)));
typedef GAS unsigned gu32;

constexpr int NRANK = 8;
constexpr unsigned MiB = 1u << 20;
constexpr unsigned XROW = 2048;
constexpr unsigned WROW = 16384;
constexpr unsigned IN_BYTES = 16 * MiB;
constexpr unsigned OUT_BYTES = 16 * MiB;
constexpr unsigned TILE_BYTES = 128 * 1024;
constexpr unsigned WS_CALL = 0;
constexpr unsigned WS_TMO = 4096;
constexpr unsigned WS_TICK = 8192;
constexpr unsigned WS_JTICK = 12288;
constexpr unsigned WS_FLAG = 16384;
constexpr unsigned ID_IN = 0;
constexpr unsigned ID_RP = 384;
constexpr unsigned ID_AV = 640;
constexpr unsigned ID_HP = 1280;
constexpr unsigned ID_DP = 1408;
constexpr unsigned WS_EX = 1 * MiB;
constexpr unsigned EX_PAR = 32 * MiB;
constexpr unsigned WS_XR = WS_EX + 64 * MiB;
constexpr unsigned XARR = 8 * MiB;
constexpr unsigned WS_WR = WS_XR + 24 * MiB;
constexpr unsigned WS_LP = WS_WR + 48 * MiB;
constexpr unsigned LP_PAR = 26 * MiB;
constexpr unsigned WS_HP = WS_LP + 52 * MiB;
constexpr unsigned WS_DP = WS_HP + 16 * MiB;
constexpr unsigned WS_RP = WS_DP + 16 * MiB;
constexpr unsigned WS_END = WS_RP + 24 * MiB;

constexpr int HTB = 16384;
constexpr int LDS_STAGE = 8 * HTB, LDS_BYTES = LDS_STAGE + 16;
constexpr unsigned long long DEADLINE_TICKS = 50000000ull;
constexpr int MAXITEMS = 96, MAXJOBS = 60, NCOMPUTE = 28, MAXZ = 96;
#define NEXPORT 4
#define NURGENT 4
constexpr int NFIRST = NCOMPUTE - NEXPORT;

struct Link { char* ws; int nitems, npush; };
struct Params {
    const char* x; const char* w; char* ws; int rank, njobs;
    char* out; int nz, zdelay; unsigned long long hops, hop3;
    Link link[NRANK];
    unsigned char zorder[MAXZ];
    unsigned short item[8][MAXITEMS];
    unsigned job[8][MAXJOBS];
};
static_assert(sizeof(Params) <= 4096, "kernel arguments");

typedef float f32x2 __attribute__((ext_vector_type(2)));
typedef __bf16 bf16x2n __attribute__((ext_vector_type(2)));
__device__ __forceinline__ unsigned cvt_pk_bf16(float lo, float hi) {
    return __builtin_bit_cast(unsigned, __builtin_convertvector((f32x2){lo, hi}, bf16x2n));
}
__device__ __forceinline__ float bf_lo(unsigned u) { return __uint_as_float(u << 16); }
__device__ __forceinline__ float bf_hi(unsigned u) { return __uint_as_float(u & 0xffff0000u); }
__device__ __forceinline__ f32x4 up_lo(u32x4 c) { return (f32x4){bf_lo(c.x), bf_hi(c.x), bf_lo(c.y), bf_hi(c.y)}; }
__device__ __forceinline__ f32x4 up_hi(u32x4 c) { return (f32x4){bf_lo(c.z), bf_hi(c.z), bf_lo(c.w), bf_hi(c.w)}; }
__device__ __forceinline__ u32x4 pack8(f32x4 a, f32x4 b) {
    return (u32x4){cvt_pk_bf16(a[0], a[1]), cvt_pk_bf16(a[2], a[3]), cvt_pk_bf16(b[0], b[1]), cvt_pk_bf16(b[2], b[3])};
}
__device__ __forceinline__ u32x4 silu_pack8(f32x4 a, f32x4 b, unsigned nan) {
    f32x4 c, d;
#pragma unroll
    for (int i = 0; i < 4; ++i) { c[i] = __builtin_amdgcn_fmed3f(a[i], -60.0f, 60.0f); d[i] = __builtin_amdgcn_fmed3f(b[i], -60.0f, 60.0f); }
    c *= -1.44269504088896341f; d *= -1.44269504088896341f;
    asm volatile("" : "+v"(c), "+v"(d));
#pragma unroll
    for (int i = 0; i < 4; ++i) { c[i] = __builtin_amdgcn_exp2f(c[i]); d[i] = __builtin_amdgcn_exp2f(d[i]); }
    asm volatile("" : "+v"(c), "+v"(d));
    c += 1.0f; d += 1.0f;
    asm volatile("" : "+v"(c), "+v"(d));
#pragma unroll
    for (int i = 0; i < 4; ++i) { c[i] = __builtin_amdgcn_rcpf(c[i]); d[i] = __builtin_amdgcn_rcpf(d[i]); }
    asm volatile("" : "+v"(c), "+v"(d));
    const u32x4 o = pack8(a * c, b * d);
    const auto sx = __builtin_amdgcn_permlane16_swap(o.x, o.z, false, false);
    const auto sy = __builtin_amdgcn_permlane16_swap(o.y, o.w, false, false);
    return (u32x4){sx[0] | nan, sy[0] | nan, sx[1] | nan, sy[1] | nan};
}
__device__ __forceinline__ unsigned ld_sys(const char* p) {
    return __hip_atomic_load((const gu32*)p, __ATOMIC_RELAXED, __HIP_MEMORY_SCOPE_SYSTEM);
}
__device__ __forceinline__ unsigned ld_agent(const char* p) {
    return __hip_atomic_load((const gu32*)p, __ATOMIC_RELAXED, __HIP_MEMORY_SCOPE_AGENT);
}
__device__ __forceinline__ unsigned grow_of(int g, int lr) {
    if (lr < 4) return (unsigned)((g ^ 1) * 1024 + lr * 256);
    if (lr >= 12) return (unsigned)(g * 1024 + (lr - 12) * 256);
    const int r = lr - 4;
    return (unsigned)((g ^ (4 + (r & 3))) * 1024 + ((g >> 1) & 1) * 512 + (r >> 2) * 256);
}
__device__ __forceinline__ unsigned khalf(int g, int j) {
    return (unsigned)((g ^ (((g ^ j) >> 2) & (g >> 1) & j)) & 1);
}
__device__ __forceinline__ unsigned in_off(int g, int j, unsigned it) {
    const unsigned blk = (it >> 1) & 15u, hh = it & 1u, kh = khalf(g, j);
    return (it & 32u) ? IN_BYTES + (kh * 512u + hh * 256u) * WROW + (unsigned)((g >> 2) & 1) * 8192u + blk * 512u
                      : (grow_of(g, (int)blk) + hh * 128u) * XROW + kh * 1024u;
}
__device__ __forceinline__ unsigned dp_off(unsigned t, unsigned par) {
    return (t < 64u ? WS_DP : WS_EX + (par ^ 1u) * EX_PAR - 64u * 2u * TILE_BYTES) + t * (2u * TILE_BYTES);
}
__device__ __forceinline__ unsigned rp_index(int j, unsigned it) {
    const unsigned lr = it >> 4;
    return lr < 4u ? it : 64u + (unsigned)(j & 1) * 64u + ((unsigned)((j >> 1) & 1) * 2u + ((lr - 4u) >> 2)) * 16u + (it & 15u);
}
__device__ __forceinline__ bool give_up(char* ws, unsigned call, unsigned long long t0);
__device__ __forceinline__ bool wave_wait(char* ws, const char* fp, unsigned call, unsigned need) {
    const unsigned long long t0 = __builtin_amdgcn_s_memrealtime();
    for (unsigned polls = 1;; ++polls) {
        const unsigned f = ld_sys(fp), tm = ld_agent(ws + WS_TMO);
        if (__builtin_amdgcn_readfirstlane(tm) == call) return true;
        if (__all(f >= need)) return false;
        if ((polls & 15u) == 0u && give_up(ws, call, t0)) return true;
        __builtin_amdgcn_s_sleep(4);
    }
}
__device__ __forceinline__ bool give_up(char* ws, unsigned call, unsigned long long t0) {
    const unsigned t = __builtin_amdgcn_readfirstlane(ld_agent(ws + WS_TMO));
    if (t == call) return true;
    if (__builtin_amdgcn_s_memrealtime() - t0 > DEADLINE_TICKS) {
        __hip_atomic_store((gu32*)(ws + WS_TMO), call, __ATOMIC_RELAXED, __HIP_MEMORY_SCOPE_AGENT);
        return true;
    }
    return false;
}


#define WAIT_V(n) asm volatile("s_waitcnt vmcnt(" #n ")" ::: "memory")
#define WAIT_L(n) asm volatile("s_waitcnt lgkmcnt(" #n ")" ::: "memory")
#define BAR asm volatile("s_barrier" ::: "memory")
#define SCHED __builtin_amdgcn_sched_barrier(0)
#define RFL(x) __builtin_amdgcn_readfirstlane(x)

#define NPUSH 4
#define TRAIL 5
#define ZDELAY 100
#define CWIN 36
#define FARSLEEP 32
#define NEARSLEEP 2
constexpr unsigned OOB = 0x40000000u;
__device__ __forceinline__ void comm_role(const Params& p, const int cid, LAS unsigned char* lds) {
    const int d = cid < 24 ? 2 * (cid >> 3) + 2 + (cid & 1) : ((cid & 2) ? 8 : 1), cix = cid < 24 ? (cid & 7) >> 1 : (cid & 1) + ((cid >> 1) & 2);
    const int wid = RFL(threadIdx.x >> 6);
    if ((unsigned)d >= (unsigned)NRANK || wid >= 2) return;
    const int lane = threadIdx.x & 63;
    volatile LAS unsigned* open = (volatile LAS unsigned*)lds;
    char* ws = p.ws; char* pb = p.link[d].ws; char* ws2 = p.link[2].ws; char* ws3 = p.link[3].ws;
    const int g = p.rank & (NRANK - 1), peer = g ^ d, nit = p.link[d].nitems, nun = nit * 4, npush = p.link[d].npush;
    const unsigned slot = (unsigned)(d - 2);
    const unsigned lane_lin = (unsigned)lane * 16u, lane_w = (unsigned)(lane >> 5) * WROW + (unsigned)(lane & 31) * 16u;
    const unsigned long long t0 = __builtin_amdgcn_s_memrealtime();
    if (wid == 1) {
        *(LAS u32x4*)(lds + lane * 16) = (u32x4){0u, 0u, 0u, 0u}; *(LAS u32x4*)(lds + 1024 + lane * 16) = (u32x4){0u, 0u, 0u, 0u};
        open[512] = 0u;
    }
    const unsigned iw0 = p.item[d][lane], iw1 = p.item[d][lane < MAXITEMS - 64 ? 64 + lane : 0];
    const unsigned call = RFL(ld_agent(ws + WS_CALL + cid * 4)) + 1u;
    WAIT_L(0);
    BAR;
    const unsigned par = call & 1u;
    gu32* tick = (gu32*)(ws + WS_TICK + d * 256 + par * 128u);
    if (threadIdx.x == 0) {
        __hip_atomic_store((gu32*)(ws + WS_CALL + cid * 4), call, __ATOMIC_RELAXED, __HIP_MEMORY_SCOPE_AGENT);
        if (cix == 0)
            __hip_atomic_store((gu32*)(ws + WS_TICK + d * 256 + (par ^ 1u) * 128u), 0u, __ATOMIC_RELAXED, __HIP_MEMORY_SCOPE_AGENT);
    }
    unsigned dsrc[2], ddst[2], dfg[2];
#pragma unroll
    for (int h = 0; h < 2; ++h) {
        const unsigned iw = h ? iw1 : iw0, kind = (iw >> 8) & 15u, tr = 192u + (iw >> 12);
        const unsigned it = (iw & 255u) | (kind == 1u ? (unsigned)((g >> 1) & 1) : 0u);
        const unsigned blk = (it >> 1) & 15u, hh = it & 1u;
        if (kind == 2u || kind == 5u) {
            const unsigned dt = rp_index(peer, it);
            dsrc[h] = WS_LP + par * LP_PAR + (kind == 2u ? it : tr) * TILE_BYTES;
            ddst[h] = (WS_RP + dt * TILE_BYTES) | 2u;
            dfg[h] = ID_RP + dt;
        } else if (kind == 4u) {
            dsrc[h] = WS_LP + par * LP_PAR + it * TILE_BYTES;
            ddst[h] = (WS_LP + par * LP_PAR + tr * TILE_BYTES) | 2u;
            dfg[h] = (ID_RP + 192u + it) | ((ID_AV + ID_RP + it) | (unsigned)((d ^ 1) & 1) << 15) << 16;
        } else if (kind == 3u) {
            const int j = ((g & 4) ^ 4) | (int)((it >> 4) & 1u) << 1 | (d & 1);
            const unsigned c = it & 15u, h2 = (unsigned)((peer >> 1) & 1), sg = (unsigned)((j ^ g) - 2), sp = (unsigned)((j ^ peer) - 2);
            dsrc[h] = WS_WR + (sp * 512u + h2 * 256u) * WROW + par * 8192u + c * 512u;
            ddst[h] = (WS_WR + (sg * 512u + h2 * 256u) * WROW + par * 8192u + c * 512u) | 1u;
            dfg[h] = ID_IN + sg * 64u + (16u + c) * 2u + h2;
        } else {
            dsrc[h] = WS_EX + par * EX_PAR + in_off(g, peer, it);
            ddst[h] = (it & 32u) ? (WS_WR + (slot * 512u + hh * 256u) * WROW + par * 8192u + blk * 512u) | 1u
                                 : WS_XR + (slot >> 1) * XARR + (blk * 256u + hh * 128u) * XROW + (slot & 1u) * 1024u;
            dfg[h] = ID_IN + slot * 64u + it;
            if (kind == 1u) dfg[h] |= ((ID_AV + ID_IN + (unsigned)((d ^ 2 ^ (peer & 1)) - 2) * 64u + it) | (unsigned)(peer & 1) << 15) << 16;
        }
    }
    if (wid == 1) {
        const auto rl = __builtin_amdgcn_make_buffer_rsrc((void*)ws, 0, WS_EX, 0x00020000);
        unsigned long long shut0 = __ballot(lane < nit), shut1 = __ballot(64 + lane < nit);
        for (unsigned polls = 1; shut0 | shut1; ++polls) {
            const int first = shut0 ? (int)__builtin_ctzll(shut0) : 64 + (int)__builtin_ctzll(shut1);
            const int c = ((lane - first) & 63) + first;
            const bool look = c < first + 8 && (((c < 64 ? shut0 : shut1) >> lane) & 1ull);
            const u32x4 r = __builtin_amdgcn_raw_buffer_load_b128(rl, look ? WS_FLAG + (ID_AV + ((c < 64 ? dfg[0] : dfg[1]) & 0xffffu)) * 16u : OOB, 0, 17);
            if (look) *(volatile LAS u32x4*)(lds + c * 16) = r;
            const bool all = look && r.x >= call && r.y >= call && r.z >= call && r.w >= call;
            shut0 &= ~__ballot(all && c < 64); shut1 &= ~__ballot(all && c >= 64);
            const int at = (int)RFL(open[512]);
            if (at >= nit || ((polls & 63u) == 0u && give_up(ws, call, t0))) break;
            if (first > at + 3) __builtin_amdgcn_s_sleep(48); else __builtin_amdgcn_s_sleep(1);
        }
        return;
    }
    const u32x4 rs = {(unsigned)(size_t)pb, (unsigned)((size_t)pb >> 32), WS_END, 0x00020000u};
    const u32x4 rd = {(unsigned)(size_t)ws, (unsigned)((size_t)ws >> 32), WS_END, 0x00020000u};
    u32x4 X[8][4];
#pragma unroll
    for (int s = 0; s < 8; ++s)
#pragma unroll
        for (int u = 0; u < 4; ++u) X[s][u] = (u32x4){0u, 0u, 0u, 0u};
    int ub = npush + cix;
    unsigned T = 0, voa = lane_lin, dof = OOB, sta = 0, ga = 0, gprev = 0;
    int fa = -1, fprev = -1;
    const unsigned one = 1u;
    static_assert(CWIN <= 8 * TRAIL - 2, "a store follows its load by TRAIL blocks of 8 accesses");
#define MV1(k, l) "s_waitcnt vmcnt(%16)\n\tbuffer_store_dwordx4 %" #k ", %10, %11, %8 offen sc0 sc1\n\ts_add_u32 %8, %8, %12\n\t" \
        "buffer_load_dwordx4 %" #l ", %13, %14, %9 offen sc0 sc1\n\ts_add_u32 %9, %9, %15\n\t"
#define PUBLISH(u, fg) do { if (lane == 0) { \
        __hip_atomic_store((gu32*)(ws + WS_FLAG + ((fg) & 0xffffu) * 16u + ((u) & 3) * 4), call, __ATOMIC_RELAXED, __HIP_MEMORY_SCOPE_SYSTEM); \
        if ((fg) >> 16) __hip_atomic_store((gu32*)((((fg) >> 31) ? ws3 : ws2) + WS_FLAG + (((fg) >> 16) & 0x7fffu) * 16u + ((u) & 3) * 4), call, __ATOMIC_RELAXED, __HIP_MEMORY_SCOPE_SYSTEM); } } while (0)
#define PICK(v) (unsigned)(itn < 64 ? __builtin_amdgcn_readlane((int)v[0], itn & 63) : __builtin_amdgcn_readlane((int)v[1], itn & 63))
    for (;;) {
        fa = RFL(fa); fprev = RFL(fprev); ub = RFL(ub);
        const int itn = ub >> 2;
        const bool live = ub < nun;
        if (lane == 0) open[512] = (unsigned)itn;
        if (!live && fa < 0) break;
        bool real = live && RFL(open[ub & 511]) >= call;
        if (live && !real && fa < 0) {
            WAIT_V(0);
            if (fprev >= 0) { PUBLISH(fprev, gprev); fprev = -1; }
            for (unsigned polls = 1; RFL(open[ub & 511]) < call; ++polls)
                if ((polls & 4095u) == 0u && give_up(ws, call, t0)) { if (lane == 0) open[512] = 999u; return; }
            real = true;
        }
        const unsigned so = PICK(dsrc), dk = PICK(ddst), fg = PICK(dfg), kind = dk & 3u;
        const unsigned strb = RFL(kind == 0u ? XROW : (kind == 1u ? 2u * WROW : 1024u));
        const unsigned uofs = (unsigned)((ub & 3) * 32) * strb;
        const unsigned vob = kind == 1u ? lane_w : lane_lin;
        unsigned sof = RFL(real ? so + uofs : OOB);
        dof = RFL(dof); sta = RFL(sta);
        if (real && lane == 0) asm volatile("global_atomic_add %0, %1, %2, off sc0" : "+v"(T) : "v"(tick), "v"(one) : "memory");
#pragma unroll
        for (int s = 0; s < 8; ++s) {
            if (s == TRAIL - 4 && fprev >= 0) { WAIT_V(33); PUBLISH(fprev, gprev); }
            if (s == TRAIL) { voa = vob; sta = strb; dof = RFL(real ? (dk & ~3u) + uofs : OOB); }
            constexpr int NB = 8;
            const int o = (s + NB - TRAIL) % NB;
            asm volatile("s_nop 4\n\t" MV1(0, 4) MV1(1, 5) MV1(2, 6) MV1(3, 7)
                : "+v"(X[o][0]), "+v"(X[o][1]), "+v"(X[o][2]), "+v"(X[o][3]), "+v"(X[s][0]), "+v"(X[s][1]), "+v"(X[s][2]), "+v"(X[s][3]), "+s"(dof), "+s"(sof)
                : "v"(voa), "s"(rd), "s"(sta), "v"(vob), "s"(rs), "s"(strb), "i"(CWIN) : "memory", "scc");
        }
        asm volatile("" : "+v"(T));
        fprev = fa; gprev = ga;
        fa = real ? ub : -1; ga = fg;
        if (real) ub = RFL((int)T) + npush + 4;
    }
    WAIT_V(0);
    if (fprev >= 0) PUBLISH(fprev, gprev);
}

__global__ __launch_bounds__(512, 2) void gemm_rs_kernel(const Params p) {
    extern __shared__ __attribute__((aligned(16))) unsigned char shm[];
    LAS unsigned char* lds = (LAS unsigned char*)shm;
    const int g = p.rank;
    char* ws = p.ws;
    const int bid = blockIdx.x;
    if (__builtin_expect(bid < 32, 1)) {
        comm_role(p, bid, lds);
    } else {
        const int tid = threadIdx.x, wid = __builtin_amdgcn_readfirstlane(tid >> 6), lane = tid & 63;
        const int xq = bid & 7;
        int jx = (bid - 32) >> 3;
#define EXPORT(n, row0, cnt, word0, nword, CALL) do { \
            const int d = 2 + (n) % 6, i = (n) / 6; \
            int gg = g, nn = p.link[d].nitems, np = p.link[d].npush; unsigned iw = ((const unsigned*)p.item[d])[i >> 1]; \
            char* pws = p.link[d].ws; const char* px = p.x; const char* pw = p.w; \
            asm volatile("" : "+s"(gg), "+s"(nn), "+s"(np), "+s"(iw), "+s"(pws), "+s"(px), "+s"(pw));         \
            iw = (iw >> (16 * (i & 1))) & 0xffffu; \
            const int peer = (gg & 7) ^ d; \
            const unsigned kind = (iw >> 8) & 15u; \
            const bool ok = i < nn && kind <= 1u; \
            const unsigned it = (iw & 255u) | (kind == 1u ? (unsigned)((peer >> 1) & 1) : 0u), off = in_off(peer, gg, it); \
            const bool isw = (it & 32u) != 0u; \
            const auto ri = __builtin_amdgcn_make_buffer_rsrc((void*)(isw ? pw : px), 0, ok ? IN_BYTES : 0u, 0x00020000); \
            const bool push = i * 4 + (word0) < np;         \
            const auto rl = __builtin_amdgcn_make_buffer_rsrc((void*)(push ? pws : ws), 0, ok ? WS_END : 0u, 0x00020000); \
            const unsigned str = isw ? 2u * WROW : XROW; \
            const unsigned vo = off + (unsigned)(row0) * str + (isw ? (unsigned)(lane >> 5) * WROW + (unsigned)(lane & 31) * 16u : (unsigned)lane * 16u); \
            u32x4 q[cnt]; \
            _Pragma("unroll") for (int k = 0; k < (cnt); ++k) q[k] = __builtin_amdgcn_raw_buffer_load_b128(ri, (vo & (IN_BYTES - 1u)) + (unsigned)k * str, 0, 0); \
            CALL; \
            _Pragma("unroll") for (int k = 0; k < (cnt); ++k) __builtin_amdgcn_raw_buffer_store_b128(q[k], rl, vo + (unsigned)k * str + \
                (push ? WS_WR + ((unsigned)(d - 2) * 512u + (it & 1u) * 256u) * WROW + (call & 1u) * 8192u + ((it >> 1) & 15u) * 512u - off : WS_EX + (call & 1u) * EX_PAR), 0, 17); \
            WAIT_V(0); \
            BAR; \
            if (ok && tid < (nword)) __hip_atomic_store((gu32*)(pws + WS_FLAG + (ID_AV + ID_IN + (unsigned)(d - 2) * 64u + it) * 16u + ((word0) + tid) * 4), call, __ATOMIC_RELAXED, __HIP_MEMORY_SCOPE_SYSTEM); \
            if (push && tid == 0) __hip_atomic_store((gu32*)(pws + WS_FLAG + (ID_IN + (unsigned)(d - 2) * 64u + it) * 16u + (word0) * 4), call, __ATOMIC_RELAXED, __HIP_MEMORY_SCOPE_SYSTEM); \
        } while (0)
        unsigned call;
        if (jx >= NURGENT) {
            const int e = (jx - NURGENT) * 8 + xq;
            EXPORT(e >> 2, (e & 3) * 32 + wid * 4, 4, e & 3, 1, call = RFL(ld_agent(ws + WS_CALL + bid * 4)) + 1u);
        } else call = RFL(ld_agent(ws + WS_CALL + bid * 4)) + 1u;
        const int wr = wid >> 2, wc = wid & 3, fr = lane & 15, fq = lane >> 4;
        const unsigned par = call & 1u;
        const int s1 = (g >> 2) & 1;
        unsigned voffA, voffB;
        { const int r = wid * 8 + (lane >> 3); voffA = (unsigned)r * XROW + (unsigned)(((lane & 7) ^ ((r >> 1) & 7)) * 16); }
        { const int r = wid * 4 + (lane >> 4); voffB = (unsigned)r * WROW + (unsigned)(((lane & 15) ^ (((r & 3) << 2) | (((r >> 3) & 1) << 1))) * 16); }
        const unsigned ldsw = (unsigned)(size_t)lds + (unsigned)wid * 1024u;
        const int aoff0 = (wr * 64 + fr) * 128 + ((fq ^ (fr >> 1)) * 16), aoff1 = aoff0 ^ 64;
        int boff0, boff1;
        { const int q = (lane & 15) >> 2, pp = lane & 3, r = 8 * fq + q;
          boff0 = 4 * HTB + r * 256 + (((wc * 4 + (pp >> 1)) ^ ((q << 2) | ((fq & 1) << 1))) * 16) + (pp & 1) * 8; boff1 = boff0 ^ 32; }
        asm volatile("" : "+v"(boff0), "+v"(boff1));
#define SA(b, h) (((b) * 2 + (h)) * HTB)
#define SB(b, h) (((b) * 2 + (h)) * HTB)
#define DMA2(rs, voff, so, l0, o0, o1) do { unsigned _t; asm volatile( \
        "s_add_u32 m0, %3, %5\n\ts_add_u32 %0, %4, %7\n\tbuffer_load_dwordx4 %1, %2, %0 offen lds\n\t" \
        "s_add_u32 m0, %3, %6\n\ts_add_u32 %0, %4, %8\n\tbuffer_load_dwordx4 %1, %2, %0 offen lds" \
        : "=&s"(_t) : "v"(voff), "s"(rs), "s"(ldsw), "s"(so), "i"(l0), "i"((l0) + 8192), "i"(o0), "i"(o1) : "memory", "scc"); } while (0)
#define STAGE_A(b, h, so, odd) DMA2(ra, voffA, so, SA(b, h), (h) * 128 * XROW + (odd) * 128, ((h) * 128 + 64) * XROW + (odd) * 128)
#define STAGE_B(b, h, so, odd) DMA2(rb, voffB, so, 4 * HTB + SB(b, h), (h) * 256 + (odd) * 64 * WROW, (h) * 256 + ((odd) * 64 + 32) * WROW)
#define LDA(dst, b, h) do { _Pragma("unroll") for (int m = 0; m < 4; ++m) { \
        dst[m][0] = *(const LAS bf16x8*)(lds + SA(b, h) + aoff0 + m * 2048); dst[m][1] = *(const LAS bf16x8*)(lds + SA(b, h) + aoff1 + m * 2048); } } while (0)
#define TR(off) __builtin_amdgcn_ds_read_tr16_b64_v4i16((LAS bf16x4*)(lds + (off)))
#define LDB(dst, b, h) do { _Pragma("unroll") for (int k = 0; k < 2; ++k) { \
        dst[0][k] = __builtin_shufflevector(TR(SB(b, h) + boff0 + k * 8192), TR(SB(b, h) + boff0 + k * 8192 + 1024), 0, 1, 2, 3, 4, 5, 6, 7); \
        dst[1][k] = __builtin_shufflevector(TR(SB(b, h) + boff1 + k * 8192), TR(SB(b, h) + boff1 + k * 8192 + 1024), 0, 1, 2, 3, 4, 5, 6, 7); } } while (0)
#define NEXTJOB() ({ if (threadIdx.x == 0 && !have) { lctl[0] = __hip_atomic_fetch_add((gu32*)(ws + WS_JTICK + xq * 256 + par * 128u), 1u, __ATOMIC_RELAXED, __HIP_MEMORY_SCOPE_AGENT) + NFIRST; WAIT_L(0); } \
        have = false; BAR; jx = RFL(lctl[0]); WAIT_L(0); BAR; })
#define WG_WAIT(fp) ({ if (wid == 0) { if (wave_wait(ws, fp, call, call) && threadIdx.x == 0) lctl[1] = 1u; WAIT_L(0); } BAR; RFL(lctl[1]) != 0u; })
#define PIN(ai, bj) asm volatile("" : "+v"(acc[ai][bj][0][0]), "+v"(acc[ai][bj][0][1]), "+v"(acc[ai][bj][1][0]), "+v"(acc[ai][bj][1][1]), \
        "+v"(acc[ai][bj][2][0]), "+v"(acc[ai][bj][2][1]), "+v"(acc[ai][bj][3][0]), "+v"(acc[ai][bj][3][1]))
#define MMA(ai, bj, At, Bt) do { __builtin_amdgcn_s_setprio(1); PIN(ai, bj); \
        _Pragma("unroll") for (int m = 0; m < 4; ++m) _Pragma("unroll") for (int n = 0; n < 2; ++n) _Pragma("unroll") for (int k = 0; k < 2; ++k) \
        acc[ai][bj][m][n] = __builtin_amdgcn_mfma_f32_16x16x32_bf16(Bt[n][k], At[m][k], acc[ai][bj][m][n], 0, 0, 0); \
        PIN(ai, bj); __builtin_amdgcn_s_setprio(0); } while (0)

        LAS unsigned* lctl = (LAS unsigned*)(lds + LDS_STAGE);
        if (tid == 0) {
            lctl[1] = 0u; lctl[3] = 0u;
            if (bid < 40) __hip_atomic_store((gu32*)(ws + WS_JTICK + xq * 256 + (par ^ 1u) * 128u), 0u, __ATOMIC_RELAXED, __HIP_MEMORY_SCOPE_AGENT);
        }
        const int njobs = p.njobs, nall = njobs + p.nz;
        bool have = false;
        if (jx >= NFIRST) {
            for (int n = 2 * (NCOMPUTE - NURGENT) + (jx - NFIRST) * 8 + xq; n < 6 * MAXITEMS; n += 8 * NEXPORT) EXPORT(n, wid * 16, 16, 0, 4, (void)0);
            NEXTJOB();
        }
        for (; jx < njobs; NEXTJOB()) {
            const unsigned job = p.job[xq][jx];
            const int type = (job >> 8) & 7, lr = (job >> 4) & 15, lc = job & 15;
            if (wid == 0) __builtin_amdgcn_fence(__ATOMIC_ACQUIRE, "");
            f32x4 acc[2][2][4][2];
#pragma unroll
            for (int i = 0; i < 32; ++i) acc[i >> 4][(i >> 3) & 1][(i >> 1) & 3][i & 1] = (f32x4){0.f, 0.f, 0.f, 0.f};
            const unsigned ma = (job >> 11) & 63u, mb = (job >> 17) & 63u, ht = (job >> 23) & 127u;
            unsigned todo = ma | mb | ((job >> 30) & 1u) << 6 | (type == 3 ? 128u : 0u) | (job >> 31) << 8;
            bool bad = false;
            while (todo) {
                const char* abase; const char* bbase; unsigned a0, b0, nbytes, segs; int nt;
                if (todo & 256u) {
                    todo &= ~256u;
                    const bool other = type == 4 && lr < 4;
                    const unsigned grow = grow_of(g, other ? lr + 12 : lr);
                    const unsigned gcol = (unsigned)((other ? 1 - s1 : s1) * 4096 + lc * 256);
                    abase = p.x; bbase = p.w; nbytes = IN_BYTES;
                    a0 = grow * XROW; b0 = gcol * 2u; nt = 16; segs = 0x10u;
                } else {
                    int tz = threadIdx.x; asm volatile("" : "+v"(tz));
                    const int el = tz & 63;
                    const unsigned t1 = (unsigned)((lr - 12) * 16 + lc);
                    if (wid == 0) {
                        const unsigned pa = todo & ma, pb = todo & mb, lead = (pa & (0u - pa)) | pb;
                        const unsigned sbit = el < 48 ? 1u << (el >> 3) : 0u;
                        const unsigned fo = (ID_IN + (unsigned)(el >> 3) * 64u) * 16u + (unsigned)(el & 7) * 4u;
                        const unsigned o0 = el < 48 ? WS_FLAG + fo + (unsigned)lr * 32u
                                          : el < 52 && (todo & 64u) ? WS_FLAG + (ID_HP + ht) * 16u + (unsigned)(el & 3) * 4u
                                          : el >= 52 && el < 56 && (todo & 128u) ? WS_FLAG + (ID_RP + t1) * 16u + (unsigned)(el & 3) * 4u : WS_TMO;
                        const unsigned o1 = WS_FLAG + fo + (unsigned)(16 + lc) * 32u;
                        const unsigned long long t0 = __builtin_amdgcn_s_memrealtime();
                        unsigned watch = pa | pb, got;
                        for (unsigned polls = 1;; ++polls) {
                            const bool on = (sbit & watch) != 0u;
                            const unsigned f0 = ld_sys(ws + (on || el >= 48 ? o0 : WS_TMO)), f1 = ld_sys(ws + (on ? o1 : WS_TMO));
                            const unsigned long long k0 = __ballot(f0 >= call && (on || el >= 48)), k1 = __ballot(f1 >= call && on);
                            unsigned full = 0u, soon = 0u;
#pragma unroll
                            for (int sl = 0; sl < 6; ++sl) {
                                const int c = __builtin_popcount((unsigned)(k0 >> (8 * sl)) & 255u) + __builtin_popcount((unsigned)(k1 >> (8 * sl)) & 255u);
                                full |= c == 16 ? 1u << sl : 0u; soon |= c >= 12 ? 1u << sl : 0u;
                            }
                            got = (full & pa) == pa ? pa : 0u;
                            if (got || !(soon & ~full & pb)) got |= full & pb;
                            got |= ((((k0 >> 48) & 15u) == 15u ? 64u : 0u) | (((k0 >> 52) & 15u) == 15u ? 128u : 0u)) & todo;
                            if (__builtin_amdgcn_readlane((int)f0, 63) == (int)call || ((polls & 15u) == 0u && give_up(ws, call, t0))) { got = 512u; break; }
                            if (got) break;
                            const bool near = !lead || (soon & lead);
                            watch = near ? pa | pb : lead;
                            if (near) __builtin_amdgcn_s_sleep(NEARSLEEP); else __builtin_amdgcn_s_sleep(FARSLEEP);
                        }
                        if (el == 0) { lctl[2] = got; if (got == 512u) lctl[1] = 1u; }
                        WAIT_L(0);
                    }
                    BAR;
                    const unsigned got = RFL(lctl[2]);
                    if (got == 512u) { bad = true; break; }
                    if (!(got & 63u)) {
                        const unsigned ad = got & (0u - got);
                        todo &= ~ad;
                        const auto rl = __builtin_amdgcn_make_buffer_rsrc((void*)ws, 0, WS_END, 0x00020000);
                        const unsigned base1 = (ad == 128u ? WS_RP + t1 * TILE_BYTES : WS_HP + ht * TILE_BYTES) + (unsigned)(wid * 16384 + el * 16);
                        u32x4 q[16];
#pragma unroll
                        for (int i = 0; i < 16; ++i) q[i] = __builtin_amdgcn_raw_buffer_load_b128(rl, base1 + (unsigned)(i * 1024), 0, 17);
                        SCHED;
#pragma unroll
                        for (int i = 0; i < 16; ++i) {
                            const int a = i >> 3, b = (i >> 2) & 1, m = i & 3;
                            acc[a][b][m][0] += up_lo(q[i]); acc[a][b][m][1] += up_hi(q[i]);
                            asm volatile("" : "+v"(acc[a][b][m][0]), "+v"(acc[a][b][m][1]));
                        }
                        if (todo) BAR;
                        continue;
                    }
                    const unsigned mask = got & 63u;
                    todo &= ~mask;
                    abase = ws; bbase = ws; nbytes = WS_END;
                    a0 = WS_XR + (unsigned)lr * 256u * XROW; b0 = WS_WR + par * 8192u + (unsigned)lc * 512u;
                    segs = 0u; nt = 0;
                    for (unsigned pr = (xq & 1) ? 0x253401u : 0x342501u, k = 0; k < 6u; ++k, pr >>= 4)
                        if ((mask >> (pr & 15u)) & 1u) { segs = segs << 4 | (pr & 15u); nt += 8; }
                }
                a0 = RFL(a0); b0 = RFL(b0); segs = RFL(segs); nt = RFL(nt);
                bf16x8 At[4][2], B0[2][2], B1[2][2];
                const u32x4 ra = {(unsigned)(size_t)abase, (unsigned)((size_t)abase >> 32), nbytes, 0x00020000u};
                const u32x4 rb = {(unsigned)(size_t)bbase, (unsigned)((size_t)bbase >> 32), nbytes, 0x00020000u};
#define KOFF(t, oa, ob) do { const int _s = (segs >> (((t) >> 3) * 4)) & 15u; \
                oa = a0 + (unsigned)(_s >> 1) * XARR + (unsigned)(_s & 1) * 1024u + (unsigned)((t) & 7) * 128u; \
                ob = b0 + (unsigned)(_s * 8 + ((t) & 7)) * (64u * WROW); } while (0)
                unsigned ca, cb;
                KOFF(0, ca, cb);
                asm volatile("s_nop 4" :: "s"(ra), "s"(rb));
                STAGE_B(0, 0, cb, 0); STAGE_A(0, 0, ca, 0); STAGE_B(0, 1, cb, 0); STAGE_A(0, 1, ca, 0);
                STAGE_B(1, 0, cb, 1); STAGE_A(1, 0, ca, 1); STAGE_B(1, 1, cb, 1);
                if (wr == 1) BAR;
                WAIT_V(10); BAR;
                WAIT_V(6); BAR;
                for (int t = 0; t < nt; t += 2) {
                    unsigned a2, b2;
                    KOFF(t + 2, a2, b2);
                    if (t + 2 >= nt) { a2 = OOB; b2 = OOB; }
                    LDB(B0, 0, 0); SCHED; LDA(At, 0, 0); STAGE_A(1, 1, ca, 1);
                    WAIT_L(8); BAR; WAIT_L(0); MMA(0, 0, At, B0); BAR; SCHED;
                    LDB(B1, 0, 1); STAGE_B(0, 0, b2, 0);
                    BAR; WAIT_L(0); MMA(0, 1, At, B1); BAR;
                    LDA(At, 0, 1); STAGE_A(0, 0, a2, 0);
                    BAR; WAIT_L(0); MMA(1, 0, At, B0); BAR; SCHED;
                    STAGE_B(0, 1, b2, 0);
                    WAIT_V(6); BAR; MMA(1, 1, At, B1); BAR;
                    LDB(B0, 1, 0); SCHED; LDA(At, 1, 0); STAGE_A(0, 1, a2, 0);
                    WAIT_L(8); BAR; WAIT_L(0); MMA(0, 0, At, B0); BAR; SCHED;
                    LDB(B1, 1, 1); STAGE_B(1, 0, b2, 1);
                    BAR; WAIT_L(0); MMA(0, 1, At, B1); BAR;
                    LDA(At, 1, 1); STAGE_A(1, 0, a2, 1);
                    BAR; WAIT_L(0); MMA(1, 0, At, B0); BAR; SCHED;
                    STAGE_B(1, 1, b2, 1);
                    WAIT_V(6); BAR; MMA(1, 1, At, B1); BAR;
                    ca = a2;
                }
                WAIT_V(0);
                if (wr == 0) BAR;
                BAR;
            }
            int tz = threadIdx.x; asm volatile("" : "+v"(tz));
            const int elane = tz & 63;
            const auto rl = __builtin_amdgcn_make_buffer_rsrc((void*)ws, 0, WS_END, 0x00020000);
            const unsigned tl = (unsigned)((lr < 12 ? lr : lr - 8) * 16 + lc);
            const unsigned pbase = (type == 2 ? WS_HP + ht * TILE_BYTES : WS_LP + par * LP_PAR + tl * TILE_BYTES) + (unsigned)(wid * 16384 + elane * 16);
            if (type == 3) {
                unsigned tk = 1u;
                if (tz == 0) asm volatile("global_atomic_add %0, %1, %0, off sc0" : "+v"(tk) : "v"(ws + WS_JTICK + xq * 256 + par * 128u) : "memory");
                const auto ro = __builtin_amdgcn_make_buffer_rsrc((void*)p.out, 0, OUT_BYTES, 0x00020000);
                const unsigned obase = (unsigned)((lr - 12) * 256 + wr * 64 + (elane & 15)) * WROW
                                     + (unsigned)(s1 * 4096 + lc * 256 + wc * 32 + ((elane >> 4) & 1) * 16 + (elane >> 5) * 8) * 2u;
#pragma unroll
                for (int i = 0; i < 16; ++i) {
                    const int a = i >> 3, b = (i >> 2) & 1, m = i & 3;
                    __builtin_amdgcn_raw_buffer_store_b128(silu_pack8(acc[a][b][m][0], acc[a][b][m][1], bad ? 0x7fc07fc0u : 0u), ro,
                                                           obase + (unsigned)(a * 128 + m * 16) * WROW + (unsigned)(b * 256), 0, 16);
                }
                asm volatile("s_waitcnt vmcnt(16)" : "+v"(tk) :: "memory");
                if (tz == 0) { lctl[0] = tk + NFIRST; WAIT_L(0); }
                have = true;
            } else if (type == 4) {
#pragma unroll
                for (int i = 0; i < 16; ++i)
#pragma unroll
                    for (int n = 0; n < 2; ++n)
                        __builtin_amdgcn_raw_buffer_store_b128(__builtin_bit_cast(u32x4, acc[i >> 3][(i >> 2) & 1][i & 3][n]), rl,
                            dp_off(tl, par) + (unsigned)wid * 32768u + (unsigned)elane * 16u + (unsigned)((i * 2 + n) * 1024), 0, 16);
            } else {
#pragma unroll
                for (int i = 0; i < 16; ++i)
                    __builtin_amdgcn_raw_buffer_store_b128(pack8(acc[i >> 3][(i >> 2) & 1][i & 3][0], acc[i >> 3][(i >> 2) & 1][i & 3][1]), rl, pbase + (unsigned)(i * 1024), 0, 17);
            }
            if (type != 3) {
                asm volatile("s_waitcnt vmcnt(0) lgkmcnt(0)" ::: "memory");
                BAR;
                if (lctl[1] == 0u) {
                    if (type == 1) {
                        const bool hop = lr < 4 && ((p.hops >> tl) & 1ull);
                        const int d = lr < 4 ? (hop ? 2 + (int)((p.hop3 >> tl) & 1ull) : 1) : 4 + ((lr - 4) & 3);
                        if (tz < 4)
                            __hip_atomic_store((gu32*)(p.link[d].ws + WS_FLAG + (ID_AV + ID_RP + (hop ? 192u + tl : rp_index(g, tl))) * 16u + tz * 4), call, __ATOMIC_RELAXED, __HIP_MEMORY_SCOPE_SYSTEM);
                    } else if (tz < 4)
                        __hip_atomic_store((gu32*)(ws + WS_FLAG + (type == 4 ? ID_DP + tl : ID_HP + ht) * 16u + tz * 4), call, __ATOMIC_RELAXED, __HIP_MEMORY_SCOPE_AGENT);
                }
            }
        }
        for (; jx < nall; NEXTJOB()) {
            const unsigned z = (unsigned)((jx - njobs) * 8 + xq), zt = p.zorder[z >> 2], t = zt & 127u, u = z & 3u;
            const bool direct = zt >= 128u;
            const int d1 = t < 64u ? ((((g & 4) ^ 4) | (int)((t >> 5) & 1u) << 1) ^ g) & 7 : 1, d2 = t < 64u ? d1 ^ 1 : 1;
            const int ow = (int)(2u * u) + (wid >> 2), grp = wid & 3;
            const auto rl = __builtin_amdgcn_make_buffer_rsrc((void*)ws, 0, WS_END, 0x00020000);
            (void)WG_WAIT(ws + WS_FLAG + (ID_DP + t) * 16u);
            const char* jt = ws + WS_JTICK + xq * 256 + par * 128u;
            u32x4 dq[8], q1[4], q2[4];
#pragma unroll
            for (int i = 0; i < 8; ++i)
                dq[i] = __builtin_amdgcn_raw_buffer_load_b128(rl, dp_off(t, par) + (unsigned)(ow * 32768 + (grp * 8 + i) * 1024 + lane * 16), 0, 17);
            if (wid == 1) {
                const unsigned long long t0 = __builtin_amdgcn_s_memrealtime();
                bool empty;
                do { empty = (int)RFL(ld_agent(jt)) + NFIRST >= nall; if (!empty) __builtin_amdgcn_s_sleep(32); }
                while (!empty && RFL(*(volatile LAS unsigned*)(lctl + 3)) != z + 1u && __builtin_amdgcn_s_memrealtime() - t0 < 2u * DEADLINE_TICKS);
                if (lane == 0) lctl[2] = empty ? 1u : 0u;
                WAIT_L(0);
            }
            if (wid == 0) {
                bool late = wave_wait(ws, ws + WS_FLAG + ((direct ? ID_AV : 0u) + ID_RP + (t < 64u ? 64u + t + (unsigned)(lane >> 5) * 64u : t - 64u)) * 16u + u * 4u, call, call);
                if (direct && !late) {
                    const int dd = lane < 32 ? d1 : d2;
                    late = wave_wait(ws, ws + WS_TICK + dd * 256 + par * 128u, call, (unsigned)(p.link[dd].nitems * 4 - p.link[dd].npush));
                    const unsigned long long t0 = __builtin_amdgcn_s_memrealtime();
                    while (__builtin_amdgcn_s_memrealtime() - t0 < (unsigned long long)p.zdelay) { }
                }
                if (lane == 0) { if (late) lctl[1] = 1u; lctl[3] = z + 1u; }
                WAIT_L(0);
            }
            BAR;
            const bool bad = RFL(lctl[1]) != 0u;
            char* b1 = p.link[d1].ws; char* b2 = p.link[d2].ws;
            asm volatile("" : "+s"(b1), "+s"(b2));
            const auto r1 = __builtin_amdgcn_make_buffer_rsrc((void*)(direct ? b1 : ws), 0, WS_END, 0x00020000);
            const auto r2 = __builtin_amdgcn_make_buffer_rsrc((void*)(direct ? b2 : ws), 0, WS_END, 0x00020000);
            const unsigned in = (unsigned)(ow * 16384 + grp * 4096 + lane * 16), lp = WS_LP + par * LP_PAR + ((t & 15u) + ((t >> 4) & 1u) * 64u) * TILE_BYTES;
            const unsigned base = bad ? WS_END : in + (!direct ? WS_RP + (t ^ 64u) * TILE_BYTES
                                                               : t < 64u ? lp + (unsigned)d1 * 16u * TILE_BYTES : WS_LP + par * LP_PAR + (t - 64u) * TILE_BYTES);
            const unsigned base2 = bad || t >= 64u ? WS_END : direct ? in + lp + (unsigned)d2 * 16u * TILE_BYTES : base + 64u * TILE_BYTES;
#pragma unroll
            for (int m = 0; m < 4; ++m) {
                q1[m] = __builtin_amdgcn_raw_buffer_load_b128(r1, base + (unsigned)(m * 1024), 0, 17);
                q2[m] = __builtin_amdgcn_raw_buffer_load_b128(r2, base2 + (unsigned)(m * 1024), 0, 17);
            }
            const auto ro = __builtin_amdgcn_make_buffer_rsrc((void*)p.out, 0, OUT_BYTES, 0x00020000);
            const unsigned obase = (((t >> 4) & 3u) * 256u + (unsigned)((grp >> 1) * 128 + (ow >> 2) * 64 + fr)) * WROW
                                 + ((unsigned)((t < 64u ? 1 - s1 : s1) * 4096 + (grp & 1) * 128 + (ow & 3) * 32 + (fq & 1) * 16 + (fq >> 1) * 8) + (t & 15u) * 256u) * 2u;
#pragma unroll
            for (int m = 0; m < 4; ++m) {
                const f32x4 v0 = __builtin_bit_cast(f32x4, dq[2 * m]) + up_lo(q1[m]) + up_lo(q2[m]);
                const f32x4 v1 = __builtin_bit_cast(f32x4, dq[2 * m + 1]) + up_hi(q1[m]) + up_hi(q2[m]);
                __builtin_amdgcn_raw_buffer_store_b128(silu_pack8(v0, v1, bad ? 0x7fc07fc0u : 0u), ro, obase + (unsigned)(m * 16) * WROW, 0, 16);
            }
            if (RFL(lctl[2]) != 0u) break;
        }
        if (tid == 0) __hip_atomic_store((gu32*)(ws + WS_CALL + bid * 4), call, __ATOMIC_RELAXED, __HIP_MEMORY_SCOPE_AGENT);
    }
}

constexpr int S_NITEMS[8] = {0, 55, 84, 84, 85, 85, 85, 85}, S_NJOBS = 54, S_NZ = 74;
constexpr unsigned short S_ITEM[8][MAXITEMS] = {
    {0},
    {
     0x233, 0x203, 0x209, 0x202, 0x239, 0x232, 0x20f, 0x23f, 0x212, 0x213, 0x219, 0x21f, 0x22f, 0x222, 0x223, 0x207, 0x217, 0x237, 0x229, 0x206,
     0x216, 0x205, 0x236, 0x215, 0x235, 0x227, 0x226, 0x20a, 0x21a, 0x23a, 0x225, 0x22a, 0x204, 0x20b, 0x214, 0x21b, 0x234, 0x23b, 0x224, 0x201,
     0x211, 0x231, 0x22b, 0x221, 0x200, 0x210, 0x230, 0x220, 0x208, 0x218, 0x22d, 0x23d, 0x20c, 0x21c, 0x22c,},
    {
     0x026, 0x027, 0x000, 0x001, 0x008, 0x009, 0x006, 0x007, 0x024, 0x025, 0x032, 0x033, 0x03e, 0x03f, 0x002, 0x003, 0x004, 0x005, 0x307, 0x317,
     0x02e, 0x02f, 0x306, 0x316, 0x02a, 0x02b, 0x028, 0x029, 0x014, 0x015, 0x034, 0x035, 0x022, 0x023, 0x02c, 0x02d, 0x016, 0x017, 0x00c, 0x00d,
     0x00a, 0x00b, 0x00e, 0x00f, 0x012, 0x013, 0x036, 0x037, 0x010, 0x011, 0x308, 0x318, 0x30e, 0x31e, 0x03a, 0x03b, 0x30c, 0x31c, 0x020, 0x021,
     0x30d, 0x31d, 0x030, 0x031, 0x03c, 0x03d, 0x01e, 0x01f, 0x038, 0x039, 0x01a, 0x01b, 0x018, 0x019, 0x01c, 0x01d, 0x428, 0x40e, 0x42e, 0x40d,
     0x538, 0x51e, 0x53e, 0x51d,},
    {
     0x026, 0x027, 0x006, 0x007, 0x000, 0x001, 0x032, 0x033, 0x024, 0x025, 0x004, 0x005, 0x002, 0x003, 0x306, 0x316, 0x03e, 0x03f, 0x00e, 0x00f,
     0x307, 0x317, 0x02e, 0x02f, 0x02a, 0x02b, 0x008, 0x009, 0x014, 0x015, 0x034, 0x035, 0x02c, 0x02d, 0x010, 0x011, 0x00a, 0x00b, 0x00c, 0x00d,
     0x016, 0x017, 0x036, 0x037, 0x012, 0x013, 0x028, 0x029, 0x03a, 0x03b, 0x308, 0x318, 0x30e, 0x31e, 0x022, 0x023, 0x30c, 0x31c, 0x020, 0x021,
     0x30d, 0x31d, 0x030, 0x031, 0x03c, 0x03d, 0x038, 0x039, 0x01c, 0x01d, 0x01e, 0x01f, 0x018, 0x019, 0x01a, 0x01b, 0x438, 0x41e, 0x43e, 0x41d,
     0x528, 0x50e, 0x52e, 0x50d,},
    {
     0x026, 0x027, 0x006, 0x007, 0x000, 0x001, 0x032, 0x033, 0x12c, 0x12e, 0x024, 0x025, 0x03e, 0x03f, 0x002, 0x003, 0x014, 0x015, 0x00a, 0x00b,
     0x016, 0x017, 0x004, 0x005, 0x00c, 0x00d, 0x012, 0x013, 0x00e, 0x00f, 0x02a, 0x02b, 0x036, 0x037, 0x028, 0x029, 0x034, 0x035, 0x01c, 0x01d,
     0x022, 0x023, 0x020, 0x021, 0x130, 0x13c, 0x018, 0x019, 0x138, 0x01a, 0x01b, 0x01e, 0x01f, 0x13a, 0x24f, 0x242, 0x243, 0x249, 0x247, 0x246,
     0x245, 0x282, 0x283, 0x286, 0x287, 0x289, 0x28f, 0x244, 0x24a, 0x285, 0x24b, 0x28b, 0x284, 0x28a, 0x241, 0x281, 0x240, 0x280, 0x24e, 0x28e,
     0x24d, 0x28d, 0x24c, 0x28c, 0x248,},
    {
     0x026, 0x027, 0x006, 0x007, 0x000, 0x001, 0x032, 0x033, 0x12c, 0x12e, 0x024, 0x025, 0x03e, 0x03f, 0x008, 0x009, 0x002, 0x003, 0x014, 0x015,
     0x016, 0x017, 0x004, 0x005, 0x00c, 0x00d, 0x010, 0x011, 0x00e, 0x00f, 0x02a, 0x02b, 0x036, 0x037, 0x028, 0x029, 0x034, 0x035, 0x01c, 0x01d,
     0x022, 0x023, 0x020, 0x021, 0x130, 0x13c, 0x018, 0x019, 0x138, 0x01a, 0x01b, 0x01e, 0x01f, 0x13a, 0x25f, 0x252, 0x253, 0x257, 0x259, 0x256,
     0x29f, 0x292, 0x293, 0x296, 0x297, 0x299, 0x255, 0x295, 0x25b, 0x29b, 0x254, 0x294, 0x25a, 0x29a, 0x291, 0x251, 0x250, 0x290, 0x25e, 0x29e,
     0x25d, 0x29d, 0x25c, 0x29c, 0x258,},
    {
     0x026, 0x027, 0x006, 0x007, 0x000, 0x001, 0x032, 0x033, 0x12c, 0x12e, 0x024, 0x025, 0x03e, 0x03f, 0x008, 0x009, 0x002, 0x003, 0x00a, 0x00b,
     0x016, 0x017, 0x004, 0x005, 0x010, 0x011, 0x012, 0x013, 0x00e, 0x00f, 0x02a, 0x02b, 0x036, 0x037, 0x028, 0x029, 0x034, 0x035, 0x01c, 0x01d,
     0x022, 0x023, 0x020, 0x021, 0x130, 0x13c, 0x018, 0x019, 0x138, 0x01a, 0x01b, 0x01e, 0x01f, 0x13a, 0x2af, 0x2a2, 0x2a3, 0x2a9, 0x2a7, 0x26f,
     0x2a6, 0x262, 0x263, 0x266, 0x267, 0x269, 0x2a5, 0x2a4, 0x265, 0x2aa, 0x26b, 0x2ab, 0x264, 0x2a1, 0x26a, 0x261, 0x260, 0x2a0, 0x26e, 0x2ae,
     0x26d, 0x2ad, 0x26c, 0x2ac, 0x268,},
    {
     0x026, 0x027, 0x006, 0x007, 0x000, 0x001, 0x032, 0x033, 0x12c, 0x12e, 0x024, 0x025, 0x03e, 0x03f, 0x008, 0x009, 0x002, 0x003, 0x014, 0x015,
     0x00a, 0x00b, 0x004, 0x005, 0x00c, 0x00d, 0x010, 0x011, 0x012, 0x013, 0x02a, 0x02b, 0x036, 0x037, 0x028, 0x029, 0x034, 0x035, 0x01c, 0x01d,
     0x022, 0x023, 0x020, 0x021, 0x130, 0x13c, 0x018, 0x019, 0x138, 0x01a, 0x01b, 0x01e, 0x01f, 0x13a, 0x2bf, 0x2b2, 0x2b3, 0x2b7, 0x2b9, 0x27f,
     0x2b6, 0x272, 0x273, 0x277, 0x279, 0x275, 0x276, 0x2b5, 0x27a, 0x27b, 0x2bb, 0x274, 0x2b4, 0x2ba, 0x2b1, 0x271, 0x270, 0x2b0, 0x27e, 0x27d,
     0x2bd, 0x2be, 0x27c, 0x2bc, 0x278,},
};
constexpr unsigned S_JOB[8][MAXJOBS] = {
    {
     0xd804610f, 0xdc04613f, 0x80000400, 0x80000430, 0x80000407, 0x80000437, 0x80000408, 0x80000438, 0x8000040f, 0x8000043f, 0xc8046107,
     0xcc046137, 0x8007c147, 0xe004414f, 0x8006e1b7, 0x1803820f, 0x1c03823f, 0x2003824f, 0x8006e177, 0x8007c187, 0x08038207, 0x0c038237,
     0x2404c27f, 0xec0421bf, 0x2805828f, 0x2c02c2bf, 0xe402217f, 0xe802418f, 0x8007c140, 0x8006e170, 0x8007c180, 0x8006e1b0, 0xc0026100,
     0xc4026130, 0x8007c148, 0x8006e178, 0x8007c188, 0x8006e1b8, 0x8007e3f0, 0x8007e3f7, 0x8007e3f8, 0xd0026108, 0xd4026138, 0x8007e3c0,
     0x8007e3c7, 0x8007e3c8, 0x00058200, 0x04058230, 0xf40463ff, 0x10058208, 0x14058238, 0xf00263cf, 0x340382ff, 0x300582cf,},
    {
     0xd884611f, 0x80000410, 0x80000420, 0x80000417, 0x80000427, 0x80000418, 0x80000428, 0x8000041f, 0x8000042f, 0xdc82612f, 0xc8846117,
     0x800761a7, 0x8007a157, 0xcc846127, 0xec8221af, 0x1883821f, 0x80076167, 0x1c85822f, 0x2c8542af, 0x8007a197, 0x08838217, 0x0c838227,
     0xe482216f, 0xe082215f, 0x2085825f, 0x2485426f, 0xe882219f, 0x8007a150, 0x80076160, 0x8007a190, 0x800761a0, 0x2885829f, 0xc0826110,
     0xc4826120, 0x8007e3e0, 0x8007e3e7, 0x8007a158, 0x80076168, 0x8007a198, 0x800761a8, 0x8007e3e8, 0x8007e3d0, 0x8007e3d7, 0x8007e3d8,
     0xd0826118, 0xd4826128, 0x00858210, 0x04858220, 0x10858218, 0x14858228, 0xf08463df, 0xf48263ef, 0x348582ef, 0x308382df,},
    {
     0xd1026109, 0xd5026139, 0x8007c149, 0x11058209, 0x15058239, 0x80000401, 0x80000431, 0x80000406, 0x80000436, 0x80000409, 0x80000439,
     0x8000040e, 0x8000043e, 0x8006e1b9, 0x8007c146, 0x8006e1b6, 0x8006e179, 0xc9026106, 0xcd026136, 0x8006e176, 0x8007c186, 0x8007c189,
     0x09058206, 0x0d058236, 0x8007c141, 0x8006e1b1, 0x8006e171, 0x8007c181, 0xc1046101, 0xc5046131, 0x01038201, 0x05038231, 0x8007e3f1,
     0x8007e3f6, 0x8007e3f9, 0x8007e3c1, 0x8007e3c6, 0x8007e3c9, 0xd902610e, 0xdd02613e, 0xe102414e, 0xe502217e, 0xe902418e, 0xed0221be,
     0xf50464fe, 0xf10264ce, 0x1905820e, 0x1d05823e, 0x2105824e, 0x2504c27e, 0x2905828e, 0x2d04c2be, 0x350382fe, 0x310582ce,},
    {
     0xd1826119, 0x80000411, 0x80000421, 0x80000416, 0x80000426, 0x80000419, 0x80000429, 0x8000041e, 0x8000042e, 0xd5826129, 0x800761a9,
     0x8007a159, 0x11858219, 0x8007a156, 0x800761a6, 0x80076166, 0x80076169, 0xc9826116, 0xcd826126, 0x15858229, 0x8007a196, 0x8007a199,
     0x09858216, 0x0d858226, 0x800761a1, 0x80076161, 0x8007a151, 0x8007a191, 0xc1846111, 0xc5846121, 0x01838211, 0x05838221, 0x8007e3e1,
     0x8007e3e6, 0x8007e3e9, 0x8007e3d1, 0x8007e3d6, 0x8007e3d9, 0xd982611e, 0xdd82612e, 0xe182215e, 0xe582216e, 0xe982219e, 0xed8221ae,
     0x1985821e, 0x1d85822e, 0x2185825e, 0x2585426e, 0x2985829e, 0x2d8542ae, 0xf18464de, 0xf58264ee, 0x358582ee, 0x318382de,},
    {
     0xc2026102, 0xc6026132, 0x8007c142, 0x80000402, 0x80000432, 0x80000405, 0x80000435, 0x8000040a, 0x8000043a, 0x8000040d, 0x8000043d,
     0x02058202, 0x06058232, 0x8006e1b2, 0x8006e172, 0x8007c145, 0xca026105, 0xce026135, 0x8006e175, 0x8007c182, 0x8007c185, 0x8006e1b5,
     0x8007c14a, 0xd202610a, 0xd602613a, 0x8006e17a, 0x8007c18a, 0x0a058205, 0x0e058235, 0x8006e1ba, 0x1205820a, 0x1605823a, 0x8007e3f2,
     0x8007e3f5, 0x8007e3fa, 0x8007e3c2, 0x8007e3c5, 0x8007e3ca, 0xda02610d, 0xde02613d, 0xe202414d, 0xe602217d, 0xea02418d, 0xee0221bd,
     0xf60463fd, 0xf20264cd, 0x1a05820d, 0x1e05823d, 0x2205824d, 0x2604c27d, 0x2a05828d, 0x2e04c2bd, 0x360382fd, 0x320582cd,},
    {
     0xc2826112, 0x80000412, 0x80000422, 0x80000415, 0x80000425, 0x8000041a, 0x8000042a, 0x8000041d, 0x8000042d, 0xc6826122, 0x800761a2,
     0x8007a152, 0x02858212, 0x80076162, 0xca826115, 0xce826125, 0x06858222, 0x800761a5, 0x8007a192, 0x8007a155, 0x80076165, 0x8007a195,
     0x800761aa, 0xd282611a, 0xd682612a, 0x0a858215, 0x0e858225, 0x8007a15a, 0x8007616a, 0x8007a19a, 0x1285821a, 0x1685822a, 0x8007e3e2,
     0x8007e3e5, 0x8007e3ea, 0x8007e3d2, 0x8007e3d5, 0x8007e3da, 0xda82611d, 0xde82612d, 0xe282215d, 0xe682216d, 0xea82219d, 0xee8221ad,
     0xf28464dd, 0xf68263ed, 0x1a85821d, 0x1e85822d, 0x2285825d, 0x2685426d, 0x2a85829d, 0x2e8542ad, 0x368582ed, 0x328382dd,},
    {
     0xc7022133, 0xc3042103, 0x8007c143, 0x0303c203, 0x0705c233, 0x80000403, 0x80000433, 0x80000404, 0x80000434, 0x8000040b, 0x8000043b,
     0x8000040c, 0x8000043c, 0x8006e1b3, 0x8006e173, 0x8007c183, 0x8007c144, 0x8007c14b, 0x8006e17b, 0x8007c18b, 0x8006e1bb, 0x8006e174,
     0x8007c184, 0x8006e1b4, 0xcb046104, 0xcf046134, 0xd302610b, 0xd702613b, 0x0b038204, 0x0f038234, 0x1305820b, 0x1705823b, 0x8007e3f3,
     0x8007e3f4, 0x8007e3fb, 0x8007e3c3, 0x8007e3c4, 0x8007e3cb, 0xdb02610c, 0xdf02613c, 0xf70464fc, 0xf30264cc, 0xe302414c, 0xe702217c,
     0xeb02418c, 0xef0221bc, 0x1b05820c, 0x1f05823c, 0x2305824c, 0x2704c27c, 0x2b05828c, 0x2f04c2bc, 0x370382fc, 0x330582cc,},
    {
     0xc3826113, 0x80000413, 0x80000423, 0x80000414, 0x80000424, 0x8000041b, 0x8000042b, 0x8000041c, 0x8000042c, 0xc7826123, 0x800761a3,
     0x8007a153, 0x03858213, 0x80076163, 0x07858223, 0x800761a4, 0x8007a193, 0x8007a15b, 0x8007616b, 0x8007a19b, 0x800761ab, 0x8007a154,
     0x80076164, 0x8007a194, 0xcb846114, 0xcf846124, 0xd382611b, 0xd782612b, 0x0b838214, 0x0f838224, 0x1385821b, 0x1785822b, 0x8007e3e3,
     0x8007e3e4, 0x8007e3eb, 0x8007e3d3, 0x8007e3d4, 0x8007e3db, 0xdb82611c, 0xdf82612c, 0xe382215c, 0xe782216c, 0xeb82219c, 0xef8221ac,
     0xf38464dc, 0xf78264ec, 0x1b85821c, 0x1f85822c, 0x2385825c, 0x2785426c, 0x2b85829c, 0x2f8542ac, 0x378582ec, 0x338382dc,},
};
constexpr unsigned char S_Z[MAXZ] = {
     15, 47, 2, 34, 18, 50, 3, 35, 19, 51, 22, 54, 7, 39, 23, 55, 9, 41, 25, 57, 6, 38, 31, 63, 5, 37, 21, 53, 11, 43, 27, 59,
     4, 36, 20, 52, 26, 58, 10, 42, 1, 33, 17, 49, 0, 32, 16, 48, 14, 46, 13, 45, 29, 61, 30, 62, 76, 78, 94, 12, 44, 92, 110, 126,
     28, 60, 108, 77, 93, 8, 40, 252, 184, 152,};
void build_schedule(Params& p) {
    p.njobs = S_NJOBS; p.nz = S_NZ / 2; p.zdelay = ZDELAY;
    for (int k = 0; k < MAXZ; ++k) p.zorder[k] = S_Z[k];
    for (int d = 0; d < 8; ++d) {
        Link& l = p.link[d];
        l.nitems = S_NITEMS[d];
        for (int k = 0; k < MAXITEMS; ++k) p.item[d][k] = S_ITEM[d][k];
        for (int k = 0; k < MAXJOBS; ++k) p.job[d][k] = S_JOB[d][k];
    }
    for (int d = 0; d < 8; ++d) {
        int n = 0;
        while (d >= 2 && n < NPUSH && (p.item[d][n >> 2] >> 5) == 1) ++n;
        p.link[d].npush = n;
    }
    p.hops = 0; p.hop3 = 0; int ntr = 0;
    for (int d = 2; d < 4; ++d)
        for (int k = 0; k < p.link[d].nitems; ++k)
            if ((p.item[d][k] >> 8) == 4) {
                const int t = p.item[d][k] & 63;
                for (int j = 0; j < p.link[d ^ 1].nitems; ++j) if (p.item[d ^ 1][j] == (5 << 8 | t)) p.item[d ^ 1][j] |= (unsigned short)(ntr << 12);
                p.item[d][k] |= (unsigned short)(ntr << 12); p.hops |= 1ull << t; if (d == 3) p.hop3 |= 1ull << t; ++ntr;
            }
}

}

extern "C" void kernel_launch(int rank, int nranks, ncclComm_t comm,
                              hipStream_t stream,
                              void* const* d_in, const int* in_sizes, int n_in,
                              void* d_out, int out_size,
                              const int* in_dtypes, int out_dtype,
                              void* const* d_out_all,
                              void* const* d_ws_all, size_t ws_bytes) {
    (void)comm; (void)in_dtypes; (void)out_dtype; (void)d_out_all;
    if (nranks != NRANK || (unsigned)rank >= (unsigned)NRANK || n_in < 2 || ws_bytes < (size_t)WS_END ||
        in_sizes[0] != 8192 * 1024 || in_sizes[1] != 1024 * 8192 || out_size != 1024 * 8192) {
        fprintf(stderr, "kernel_launch: unexpected problem shape\n");
        return;
    }
    HIP_CHECK(hipFuncSetAttribute((const void*)gemm_rs_kernel, hipFuncAttributeMaxDynamicSharedMemorySize, LDS_BYTES));
    Params p;
    p.x = (const char*)d_in[0]; p.w = (const char*)d_in[1]; p.out = (char*)d_out;
    for (int i = 0; i < NRANK; ++i) p.link[i].ws = (char*)d_ws_all[rank ^ i];
    p.ws = p.link[0].ws;
    p.rank = rank;
    build_schedule(p);
    hipLaunchKernelGGL(gemm_rs_kernel, dim3(256), dim3(512), LDS_BYTES, stream, p);
    HIP_CHECK(hipGetLastError());
}
```
